```python
import numpy as np

kernel_name = "gpu_ag_gemm_m2048_k2048_n2048_bf16_relu"


ABSMAX_RTOL = 0.008

COMPUTE_DTYPE = "bfloat16"
EPILOGUE = "relu"
IN_QUANT = None
OUT_QUANT = None
IN_DTYPES = {"x": "bfloat16", "w_mat": "bfloat16"}
WORLD_SIZE = 8

def _to_bf16(x):
    u = np.ascontiguousarray(x, dtype=np.float32).view(np.uint32)
    return ((u + 0x7FFF + ((u >> 16) & 1)) & 0xFFFF0000).view(np.float32)

def setup_inputs():

    x = _to_bf16(np.random.randn(2048, 2048).astype(np.float32))
    w_mat = _to_bf16((np.random.randn(2048, 2048) / np.sqrt(2048)).astype(np.float32))
    return {"x": x, "w_mat": w_mat}

def _epilogue(y):
    return np.maximum(y, 0.0).astype(np.float32)

def reference(x, w_mat):
    y = (x @ w_mat).astype(np.float32)
    return _epilogue(y)
```

```cpp
#include <hip/hip_runtime.h>
#include <rccl/rccl.h>
#include <cstdio>

#define HIP_CHECK(expr) do { hipError_t _e = (expr); if (_e != hipSuccess) { \
    fprintf(stderr, "HIP error %s at %s:%d\n", hipGetErrorString(_e), __FILE__, __LINE__); \
    return; } } while (0)

namespace {

typedef unsigned u32;
typedef unsigned long long u64;
typedef u32 u32x4 __attribute__((ext_vector_type(4)));
typedef __bf16 bf16x8 __attribute__((ext_vector_type(8)));
typedef float f32x4 __attribute__((ext_vector_type(4)));

constexpr int NR = 8;
constexpr u32 KIB = 1024u;
constexpr u32 PAR_BYTES = 8u << 20;
constexpr u32 W_SLOT = 256u * KIB;
constexpr u32 XP_OFF = 4u << 20;
constexpr u32 XE_OFF = 5u << 20;
constexpr u32 X_RT = 128u * KIB;
constexpr u32 FLAG_OFF = 16u << 20;
constexpr u32 CTR_OFF = 0;
constexpr u32 TMO_OFF = 32u * KIB;
constexpr u32 WF_OFF = 36u * KIB;
constexpr u32 XPF_OFF = 40u * KIB;
constexpr u32 XEF_OFF = 44u * KIB;
constexpr u32 OF_OFF = 48u * KIB;
constexpr u32 DBG_OFF = 256u * KIB;
constexpr u32 FLAG_BYTES = 1u << 20;
constexpr u32 SHARD_BYTES = 1u << 20;
constexpr u64 DEADLINE = 100000000ull;
constexpr int AUX_SYS = 17;
constexpr int LDS_ROW = 68;

struct Params {
  char* ws[NR];
  char* out[NR];
  const char* x;
  const char* w;
  int rank;
};

__device__ __forceinline__ u64 now() { return __builtin_amdgcn_s_memrealtime(); }
__device__ __forceinline__ int par(int v) { return __builtin_popcount(v) & 1; }
__device__ __forceinline__ int fmask(int t) { return t == 3 ? 7 : (1 << t); }
__device__ __forceinline__ int ker_index(int t, int d) {
  return t == 0 ? (d >> 1) : t == 1 ? ((d & 1) | ((d >> 2) << 1)) : t == 2 ? (d & 3) : (d >> 1);
}
__device__ __forceinline__ int ker_elem(int t, int j) {
  return t == 0 ? (j << 1) : t == 1 ? ((j & 1) | ((j >> 1) << 2)) : t == 2 ? j : ((j << 1) | par(j));
}
__device__ __forceinline__ int pair_of(int d) { return (d == 1 || d == 6) ? 0 : (d == 2 || d == 5) ? 1 : 2; }
__device__ __forceinline__ __amdgpu_buffer_rsrc_t rsrc(const void* p, u32 bytes) {
  return __builtin_amdgcn_make_buffer_rsrc(const_cast<void*>(p), 0, (int)bytes, 0x00020000);
}
__device__ __forceinline__ u32 ld_sys(const u32* p) {
  return __hip_atomic_load(p, __ATOMIC_RELAXED, __HIP_MEMORY_SCOPE_SYSTEM);
}
__device__ __forceinline__ void st_sys(u32* p, u32 v) {
  __hip_atomic_store(p, v, __ATOMIC_RELAXED, __HIP_MEMORY_SCOPE_SYSTEM);
}
__device__ __forceinline__ void drain() { asm volatile("s_waitcnt vmcnt(0)" ::: "memory"); }

#define STAMP(i) do { } while (0)

struct Waiter {
  u32* tmo;
  u32 call;
  u64 t0;
  u32 spins;
  bool dead;
  __device__ __forceinline__ bool expired() {
    if (dead) return true;
    ++spins;
    if ((spins & 15u) == 0u) {
      u32 t = __hip_atomic_load(tmo, __ATOMIC_RELAXED, __HIP_MEMORY_SCOPE_AGENT);
      t = __builtin_amdgcn_readfirstlane(t);
      if (t == call) dead = true;
      else if (now() - t0 > DEADLINE) {
        __hip_atomic_store(tmo, call, __ATOMIC_RELAXED, __HIP_MEMORY_SCOPE_AGENT);
        dead = true;
      }
    }
    return dead;
  }
};

__global__ void __launch_bounds__(320) ag_gemm_kernel(Params p) {
  __shared__ __attribute__((aligned(16))) float red[4 * 32 * LDS_ROW];
  const int lane = threadIdx.x & 63;
  const int wv = __builtin_amdgcn_readfirstlane(threadIdx.x >> 6);
  const int c = blockIdx.x;
  const int r = p.rank;
  char* myws = p.ws[r];
  u32* ctr = (u32*)(myws + FLAG_OFF + CTR_OFF + (u32)c * 128u);
  u32* tmo = (u32*)(myws + FLAG_OFF + TMO_OFF);
  STAMP(0);
  const u32 call = __builtin_amdgcn_readfirstlane(
                       __hip_atomic_load(ctr, __ATOMIC_RELAXED, __HIP_MEMORY_SCOPE_AGENT)) + 1u;
  const u32 parb = (call & 1u) * PAR_BYTES;
  const int g = lane >> 4, l16 = lane & 15;

  if (wv == 4) {
    const int L = (c + (c >> 3)) & 7;
    const int idx = c >> 3;
    const int sub = idx & 1, kr = idx >> 1;
    const int dst = r ^ L;
    char* dws = p.ws[dst];
    const __amdgpu_buffer_rsrc_t dr = rsrc(dws + parb, PAR_BYTES);
    u32* dflag = (u32*)(dws + FLAG_OFF);
    const __amdgpu_buffer_rsrc_t xr = rsrc(p.x, SHARD_BYTES);
    const __amdgpu_buffer_rsrc_t wr = rsrc(p.w, SHARD_BYTES);
    const int nw = L == 0 ? 4 : L == 7 ? 0 : 2;
    const int nx = L == 0 ? 0 : L == 7 ? 5 : 1;
    int tprev = -1;
    for (int rd = 0; rd < nw; ++rd) {
      int t;
      if (L == 0) t = rd;
      else { t = tprev + 1; while (par(fmask(t) & L)) ++t; }
      tprev = t;
      const int slot = t * 4 + ker_index(t, L);
      const u32 so = (u32)(128 * kr + 8 * g) * 512u + (u32)(64 * t + 32 * sub + 2 * l16) * 2u;
      u32 v[32];
#pragma unroll
      for (int i = 0; i < 32; ++i)
        v[i] = __builtin_amdgcn_raw_buffer_load_b32(wr, so + (u32)((i >> 3) * 32 + (i & 7)) * 512u, 0, 0);
      const u32 dofs = (u32)slot * W_SLOT + (u32)(kr * 2 + sub) * 8192u + (u32)lane * 16u;
#pragma unroll
      for (int ks = 0; ks < 4; ++ks) {
        u32x4 e, o;
#pragma unroll
        for (int m = 0; m < 4; ++m) {
          e[m] = __builtin_amdgcn_perm(v[ks * 8 + 2 * m + 1], v[ks * 8 + 2 * m], 0x05040100u);
          o[m] = __builtin_amdgcn_perm(v[ks * 8 + 2 * m + 1], v[ks * 8 + 2 * m], 0x07060302u);
        }
        __builtin_amdgcn_raw_buffer_store_b128(e, dr, dofs + (u32)(ks * 2) * 1024u, 0, AUX_SYS);
        __builtin_amdgcn_raw_buffer_store_b128(o, dr, dofs + (u32)(ks * 2 + 1) * 1024u, 0, AUX_SYS);
      }
      drain();
      if (lane == 0) st_sys(dflag + (WF_OFF >> 2) + slot * 32 + kr * 2 + sub, call);
      STAMP(1 + rd);
    }
    for (int rd = 0; rd < nx; ++rd) {
      const int rt = L == 7 ? rd : 5 + pair_of(L);
      const u32 so = (u32)(32 * rt + 16 * sub + l16) * 4096u + (u32)(128 * kr + 8 * g) * 2u;
      u32x4 v[4];
#pragma unroll
      for (int ks = 0; ks < 4; ++ks) v[ks] = __builtin_amdgcn_raw_buffer_load_b128(xr, so + (u32)ks * 64u, 0, 0);
      const u32 base = L == 7 ? XP_OFF + (u32)rd * X_RT : XE_OFF + (u32)(L - 1) * X_RT;
      const u32 dofs = base + (u32)kr * 8192u + (u32)sub * 4096u + (u32)lane * 16u;
#pragma unroll
      for (int ks = 0; ks < 4; ++ks)
        __builtin_amdgcn_raw_buffer_store_b128(v[ks], dr, dofs + (u32)ks * 1024u, 0, AUX_SYS);
      drain();
      const u32 fo = L == 7 ? (XPF_OFF >> 2) + (u32)rd * 32u : (XEF_OFF >> 2) + (u32)(L - 1) * 32u;
      if (lane == 0) st_sys(dflag + fo + kr * 2 + sub, call);
      STAMP(5 + rd);
    }
    __builtin_amdgcn_s_barrier();
    return;
  }

  const int xcd = c & 7, li = c >> 3;
  const int ss = 2 * xcd + (li >> 4);
  const int ti = li & 15;
  const int t = ss >> 2;
  const int s = r ^ ker_elem(t, ss & 3);
  int prow, rt;
  u32 x_base, x_lane, x_rb, x_ks, x_kr;
  const u32* xflag = nullptr;
  __amdgpu_buffer_rsrc_t xr;
  const u32* myflag = (const u32*)(myws + FLAG_OFF);
  if (ti < 8) {
    prow = r; rt = ti;
    xr = rsrc(p.x, SHARD_BYTES);
    x_base = (u32)rt * X_RT; x_lane = (u32)l16 * 4096u + (u32)g * 16u; x_rb = 65536u; x_ks = 64u; x_kr = 256u;
  } else {
    xr = rsrc(myws + parb, PAR_BYTES);
    x_lane = (u32)lane * 16u; x_rb = 4096u; x_ks = 1024u; x_kr = 8192u;
    if (ti < 13) {
      prow = r ^ 7; rt = ti - 8;
      x_base = XP_OFF + (u32)rt * X_RT;
      xflag = myflag + (XPF_OFF >> 2) + rt * 32;
    } else {
      int e = 0, n = ti - 13;
      for (int q = 1; q <= 6; ++q) if (par(fmask(t) & q)) { if (n == 0) { e = q; } --n; }
      prow = r ^ e; rt = 5 + pair_of(e);
      x_base = XE_OFF + (u32)(e - 1) * X_RT;
      xflag = myflag + (XEF_OFF >> 2) + (e - 1) * 32;
    }
  }
  const __amdgpu_buffer_rsrc_t wr = rsrc(myws + parb + (u32)ss * W_SLOT, W_SLOT);
  const u32* wflag = myflag + (WF_OFF >> 2) + ss * 32;

  const u32* pollp = nullptr;
  if (lane < 8) { if (xflag) pollp = xflag + (4 * (lane >> 1) + wv) * 2 + (lane & 1); }
  else if (lane < 16) pollp = wflag + (4 * ((lane - 8) >> 1) + wv) * 2 + (lane & 1);

  Waiter wt; wt.tmo = tmo; wt.call = call; wt.t0 = now(); wt.spins = 0; wt.dead = false;
  u32 ready = 0;

  f32x4 acc[2][4];
#pragma unroll
  for (int a = 0; a < 2; ++a)
#pragma unroll
    for (int b = 0; b < 4; ++b) acc[a][b] = f32x4{0.f, 0.f, 0.f, 0.f};
  u32x4 xb[2][8], wb[2][16];

#define WAIT_FOR(mask)                                                        \
  while ((ready & (mask)) != (mask)) {                                        \
    u32 v_ = call;                                                            \
    if (pollp) v_ = ld_sys(pollp);                                            \
    ready = (u32)__builtin_amdgcn_ballot_w64((int)(v_ - call) >= 0);          \
    if ((ready & (mask)) == (mask)) break;                                    \
    if (wt.expired()) { ready = 0xffffffffu; break; }                         \
    __builtin_amdgcn_s_sleep(1);                                              \
  }
#define ISSUE(step)                                                           \
  {                                                                           \
    const u32 kr_ = 4 * (step) + wv;                                          \
    WAIT_FOR(0x3u << (2 * (step)));                                           \
    _Pragma("unroll") for (int i = 0; i < 8; ++i)                             \
      xb[(step) & 1][i] = __builtin_amdgcn_raw_buffer_load_b128(              \
          xr, x_base + x_lane + kr_ * x_kr + (u32)(i >> 2) * x_rb + (u32)(i & 3) * x_ks, 0, AUX_SYS); \
    WAIT_FOR(0x300u << (2 * (step)));                                         \
    _Pragma("unroll") for (int i = 0; i < 16; ++i)                            \
      wb[(step) & 1][i] = __builtin_amdgcn_raw_buffer_load_b128(              \
          wr, (u32)lane * 16u + kr_ * 16384u + (u32)i * 1024u, 0, AUX_SYS);   \
  }
#define COMPUTE(step)                                                         \
  {                                                                           \
    _Pragma("unroll") for (int ks = 0; ks < 4; ++ks)                          \
    _Pragma("unroll") for (int rb = 0; rb < 2; ++rb)                          \
    _Pragma("unroll") for (int cf = 0; cf < 4; ++cf)                          \
      acc[rb][cf] = __builtin_amdgcn_mfma_f32_16x16x32_bf16(                  \
          __builtin_bit_cast(bf16x8, wb[(step) & 1][(cf >> 1) * 8 + ks * 2 + (cf & 1)]), \
          __builtin_bit_cast(bf16x8, xb[(step) & 1][rb * 4 + ks]), acc[rb][cf], 0, 0, 0); \
  }

  ISSUE(0);
  STAMP(1);
  ISSUE(1);
  COMPUTE(0);
  STAMP(2);
  ISSUE(2);
  COMPUTE(1);
  STAMP(3);
  ISSUE(3);
  COMPUTE(2);
  STAMP(4);
  COMPUTE(3);
  STAMP(5);

  {
    float* my = red + wv * 32 * LDS_ROW;
#pragma unroll
    for (int rb = 0; rb < 2; ++rb)
#pragma unroll
      for (int cb = 0; cb < 2; ++cb) {
        float* q = my + (16 * rb + l16) * LDS_ROW + 32 * cb + 8 * g;
        const f32x4 e = acc[rb][cb * 2], o = acc[rb][cb * 2 + 1];
        *(f32x4*)q = f32x4{e[0], o[0], e[1], o[1]};
        *(f32x4*)(q + 4) = f32x4{e[2], o[2], e[3], o[3]};
      }
  }
  __syncthreads();
  STAMP(6);
  if (threadIdx.x == 0) __hip_atomic_store(ctr, call, __ATOMIC_RELAXED, __HIP_MEMORY_SCOPE_AGENT);
  {
    const int row = 8 * wv + (lane >> 3), col = (lane & 7) * 8;
    const float* q = red + row * LDS_ROW + col;
    f32x4 a = *(const f32x4*)q, b = *(const f32x4*)(q + 4);
#pragma unroll
    for (int w2 = 1; w2 < 4; ++w2) {
      a += *(const f32x4*)(q + w2 * 32 * LDS_ROW);
      b += *(const f32x4*)(q + w2 * 32 * LDS_ROW + 4);
    }
    float y[8] = {a[0], a[1], a[2], a[3], b[0], b[1], b[2], b[3]};
    u32 h[8];
#pragma unroll
    for (int i = 0; i < 8; ++i) {
      const float v = y[i] < 0.f ? 0.f : y[i];
      const u32 u = __builtin_bit_cast(u32, v);
      h[i] = (v != v) ? 0x7fc0u : ((u + 0x7fffu + ((u >> 16) & 1u)) >> 16);
    }
    u32 tv = __hip_atomic_load(tmo, __ATOMIC_RELAXED, __HIP_MEMORY_SCOPE_AGENT);
    u32x4 o;
    if (tv == call) o = u32x4{0x7fc07fc0u, 0x7fc07fc0u, 0x7fc07fc0u, 0x7fc07fc0u};
    else o = u32x4{h[0] | (h[1] << 16), h[2] | (h[3] << 16), h[4] | (h[5] << 16), h[6] | (h[7] << 16)};
    const __amdgpu_buffer_rsrc_t orr = rsrc(p.out[s], SHARD_BYTES);
    const u32 oo = (u32)(256 * prow + 32 * rt + row) * 512u + (u32)(64 * t + col) * 2u;
    __builtin_amdgcn_raw_buffer_store_b128(o, orr, oo, 0, AUX_SYS);
  }
  if (s != r) {
    drain();
    __syncthreads();
    if (threadIdx.x == 0)
      st_sys((u32*)(p.ws[s] + FLAG_OFF + OF_OFF) + (8 * prow + rt) * 4 + t, call);
  }
  STAMP(7);
  if (c == 0 && wv == 0) {
    bool done = false;
    while (!done) {
      bool ok = true;
#pragma unroll
      for (int i = 0; i < 4; ++i) {
        const int f = i * 64 + lane;
        const int ft = f & 3, frt = (f >> 2) & 7, fp = f >> 5;
        const int fd = fp ^ r;
        const bool local = par(fmask(ft) & fd) == 0 ? (fd == 0)
                           : frt < 5 ? (fd == 7) : (fd != 7 && pair_of(fd) == frt - 5);
        if (!local) {
          const u32 v = ld_sys(myflag + (OF_OFF >> 2) + f);
          ok = ok && ((int)(v - call) >= 0);
        }
      }
      done = __all(ok);
      if (!done) {
        if (wt.expired()) {
          if (lane == 0) *(unsigned short*)(p.out[r] + (size_t)(256 * r) * 512) = 0x7fc0;
          break;
        }
        __builtin_amdgcn_s_sleep(1);
      }
    }
    STAMP(8);
  }
}

}

extern "C" void kernel_launch(int rank, int nranks, ncclComm_t comm,
                              hipStream_t stream,
                              void* const* d_in, const int* in_sizes, int n_in,
                              void* d_out, int out_size,
                              const int* in_dtypes, int out_dtype,
                              void* const* d_out_all,
                              void* const* d_ws_all, size_t ws_bytes) {
  (void)comm; (void)in_dtypes; (void)out_dtype; (void)d_out;
  if (nranks != NR || n_in != 2 || in_sizes[0] != 256 * 2048 || in_sizes[1] != 2048 * 256 ||
      out_size != 2048 * 256 || ws_bytes < (size_t)FLAG_OFF + FLAG_BYTES) {
    fprintf(stderr, "kernel_launch: unexpected problem shape\n");
    return;
  }
  Params p;
  for (int i = 0; i < NR; ++i) { p.ws[i] = (char*)d_ws_all[i]; p.out[i] = (char*)d_out_all[i]; }
  p.x = (const char*)d_in[0];
  p.w = (const char*)d_in[1];
  p.rank = rank;
  ag_gemm_kernel<<<256, 320, 0, stream>>>(p);
  HIP_CHECK(hipGetLastError());
}
```

```cpp
#include <hip/hip_runtime.h>
#include <rccl/rccl.h>
#include <cstdio>

#define HIP_CHECK(expr) do { hipError_t _e = (expr); if (_e != hipSuccess) { \
    fprintf(stderr, "HIP error %s at %s:%d\n", hipGetErrorString(_e), __FILE__, __LINE__); \
    return; } } while (0)

namespace {

typedef unsigned u32;
typedef unsigned long long u64;
typedef u32 u32x4 __attribute__((ext_vector_type(4)));
typedef __bf16 bf16x8 __attribute__((ext_vector_type(8)));
typedef float f32x4 __attribute__((ext_vector_type(4)));
typedef unsigned short u16x2 __attribute__((ext_vector_type(2)));

constexpr int NR = 8;
constexpr u32 KIB = 1024u;
constexpr u32 W_SLOT = 256u * KIB;
constexpr u32 X_RT = 128u * KIB;
constexpr u32 XP_OFF = 12u * W_SLOT;
constexpr u32 XE_OFF = XP_OFF + 5u * X_RT;
constexpr u32 BUF_BYTES = XE_OFF + 6u * X_RT;
constexpr u32 OWNW_OFF = 3u * BUF_BYTES;
constexpr u32 OUTS_OFF = OWNW_OFF + 4u * W_SLOT;
constexpr u32 POISON_CHUNKS = BUF_BYTES / 16u;
constexpr u32 FLAG_OFF = 16u << 20;
constexpr u32 CTR_OFF = 0;
constexpr u32 TMO_OFF = 32u * KIB;
constexpr u32 LNK_OFF = 48u * KIB;
constexpr u32 WF_OFF = 36u * KIB;
constexpr u32 XPF_OFF = 40u * KIB;
constexpr u32 XEF_OFF = 44u * KIB;
constexpr u32 DBG_OFF = 256u * KIB;
constexpr u32 FLAG_BYTES = 1u << 20;
constexpr u32 SHARD_BYTES = 1u << 20;
constexpr u64 DEADLINE = 100000000ull;
constexpr int AUX_SYS = 17;
#define POISON_AUX 17
#define PACE 2
#define PROBE_MASK 3
#define POLL_SLEEP 1
constexpr int LDS_ROW = 68;

struct Params {
  char* ws[NR];
  char* out[NR];
  const char* x;
  const char* w;
  int rank;
};

__device__ __forceinline__ u64 now() { return __builtin_amdgcn_s_memrealtime(); }
__device__ __forceinline__ int par(int v) { return __builtin_popcount(v) & 1; }
__device__ __forceinline__ int fmask(int t) { return t == 3 ? 7 : (1 << t); }
__device__ __forceinline__ int ker_index(int t, int d) {
  return t == 0 ? (d >> 1) : t == 1 ? ((d & 1) | ((d >> 2) << 1)) : t == 2 ? (d & 3) : (d >> 1);
}
__device__ __forceinline__ int ker_elem(int t, int j) {
  return t == 0 ? (j << 1) : t == 1 ? ((j & 1) | ((j >> 1) << 2)) : t == 2 ? j : ((j << 1) | par(j));
}
__device__ __forceinline__ int pair_of(int d) { return (d == 1 || d == 6) ? 0 : (d == 2 || d == 5) ? 1 : 2; }
__device__ __forceinline__ __amdgpu_buffer_rsrc_t rsrc(const void* p, u32 bytes) {
  return __builtin_amdgcn_make_buffer_rsrc(const_cast<void*>(p), 0, (int)bytes, 0x00020000);
}
__device__ __forceinline__ u32 ld_sys(const u32* p) {
  return __hip_atomic_load(p, __ATOMIC_RELAXED, __HIP_MEMORY_SCOPE_SYSTEM);
}
__device__ __forceinline__ void st_sys(u32* p, u32 v) {
  __hip_atomic_store(p, v, __ATOMIC_RELAXED, __HIP_MEMORY_SCOPE_SYSTEM);
}
__device__ __forceinline__ void drain() { asm volatile("s_waitcnt vmcnt(0)" ::: "memory"); }
__device__ __forceinline__ u32 pk_min(u32 a, u32 b) {
  return __builtin_bit_cast(u32, __builtin_elementwise_min(__builtin_bit_cast(u16x2, a), __builtin_bit_cast(u16x2, b)));
}
__device__ __forceinline__ u32 pk_max(u32 a, u32 b) {
  return __builtin_bit_cast(u32, __builtin_elementwise_max(__builtin_bit_cast(u16x2, a), __builtin_bit_cast(u16x2, b)));
}
__device__ __forceinline__ u32 unpoison(u32 v) { return pk_min(v, 0xfffefffeu); }

#define STAMP(i) do { } while (0)

struct Waiter {
  u32* tmo;
  u32 call;
  u64 t0;
  u32 spins;
  bool dead;
  __device__ __forceinline__ bool expired() {
    if (dead) return true;
    ++spins;
    if ((spins & 15u) == 0u) {
      u32 t = __hip_atomic_load(tmo, __ATOMIC_RELAXED, __HIP_MEMORY_SCOPE_AGENT);
      t = __builtin_amdgcn_readfirstlane(t);
      if (t == call) dead = true;
      else if (now() - t0 > DEADLINE) {
        __hip_atomic_store(tmo, call, __ATOMIC_RELAXED, __HIP_MEMORY_SCOPE_AGENT);
        dead = true;
      }
    }
    return dead;
  }
};

__global__ void __launch_bounds__(320) ag_gemm_kernel(Params p) {
  __shared__ __attribute__((aligned(16))) float red[4 * 32 * LDS_ROW];
  __shared__ u32 deadf[4];
  const int lane = threadIdx.x & 63;
  const int wv = __builtin_amdgcn_readfirstlane(threadIdx.x >> 6);
  const int c = blockIdx.x;
  const int r = p.rank;
  char* myws = p.ws[r];
  u32* ctr = (u32*)(myws + FLAG_OFF + CTR_OFF + (u32)c * 128u);
  u32* tmo = (u32*)(myws + FLAG_OFF + TMO_OFF);
  STAMP(0);
  const u32 ctrv = __hip_atomic_load(ctr, __ATOMIC_RELAXED, __HIP_MEMORY_SCOPE_AGENT);
  const int g = lane >> 4, l16 = lane & 15;
#define GET_CALL()                                                             \
  const u32 call = __builtin_amdgcn_readfirstlane(ctrv) + 1u;                  \
  const u32 parb = (call % 3u) * BUF_BYTES;                                    \
  const bool slow = call <= 2u;

  if (wv == 4) {
    const int L = (c + (c >> 3)) & 7;
    const int idx = c >> 3;
    const int sub = idx & 1, kr = idx >> 1;
    const int dst = r ^ L;
    char* dws = p.ws[dst];
    u32* dflag = (u32*)(dws + FLAG_OFF);
    const __amdgpu_buffer_rsrc_t xr = rsrc(p.x, SHARD_BYTES);
    const __amdgpu_buffer_rsrc_t wr = rsrc(p.w, SHARD_BYTES);
    __builtin_amdgcn_s_setprio(3);
    const u32 w_lane = (u32)(8 * g) * 512u + (u32)(2 * l16) * 2u;
    const u32 x_lane = (u32)l16 * 4096u + (u32)(8 * g) * 2u;
#define LOAD_WH(v, t, cb, kr_, hf)                                             \
    _Pragma("unroll") for (int i = 0; i < 16; ++i)                             \
      v[i] = __builtin_amdgcn_raw_buffer_load_b32(wr, w_lane + (u32)(t) * 128u + (u32)(cb) * 64u + \
                 (u32)(128 * (kr_) + 64 * (hf) + (i >> 3) * 32 + (i & 7)) * 512u, 0, 0);
#define STORE_WH(v, off)                                                       \
    _Pragma("unroll") for (int ks = 0; ks < 2; ++ks) {                         \
      u32x4 e, o;                                                              \
      _Pragma("unroll") for (int m = 0; m < 4; ++m) {                          \
        e[m] = unpoison(__builtin_amdgcn_perm(v[ks * 8 + 2 * m + 1], v[ks * 8 + 2 * m], 0x05040100u)); \
        o[m] = unpoison(__builtin_amdgcn_perm(v[ks * 8 + 2 * m + 1], v[ks * 8 + 2 * m], 0x07060302u)); \
      }                                                                        \
      __builtin_amdgcn_raw_buffer_store_b128(e, dr, (u32)(off) + (u32)lane * 16u + (u32)(ks * 2) * 1024u, 0, AUX_SYS); \
      __builtin_amdgcn_raw_buffer_store_b128(o, dr, (u32)(off) + (u32)lane * 16u + (u32)(ks * 2 + 1) * 1024u, 0, AUX_SYS); \
    }
#define LOAD_XH(v, rt, rb, kr_)                                                \
    _Pragma("unroll") for (int ks = 0; ks < 4; ++ks) {                         \
      const u32x4 q_ = __builtin_amdgcn_raw_buffer_load_b128(xr, x_lane + (u32)(rt) * X_RT + (u32)(rb) * 65536u + (u32)(kr_) * 256u + (u32)ks * 64u, 0, 0); \
      v[ks * 4] = q_[0]; v[ks * 4 + 1] = q_[1]; v[ks * 4 + 2] = q_[2]; v[ks * 4 + 3] = q_[3]; \
    }
#define STORE_XH(v, off)                                                       \
    _Pragma("unroll") for (int ks = 0; ks < 4; ++ks)                           \
      __builtin_amdgcn_raw_buffer_store_b128(u32x4{unpoison(v[ks * 4]), unpoison(v[ks * 4 + 1]), unpoison(v[ks * 4 + 2]), unpoison(v[ks * 4 + 3])}, dr, (u32)(off) + (u32)lane * 16u + (u32)ks * 1024u, 0, AUX_SYS);
#define PUSH_SETUP()                                                           \
    __builtin_amdgcn_sched_barrier(0);                                         \
    GET_CALL();                                                                \
    STAMP(10);                                                                 \
    const __amdgpu_buffer_rsrc_t dr = rsrc(dws + (L == 0 ? OWNW_OFF : parb), BUF_BYTES);
#define WAITCNT_(n) asm volatile("s_waitcnt vmcnt(" #n ")" ::: "memory")
#define WAITCNT(n) do { if ((n) == 0) WAITCNT_(0); else if ((n) == 4) WAITCNT_(4); else if ((n) == 8) WAITCNT_(8); else WAITCNT_(12); } while (0)
    if (L == 0) {
      u32 v[8][16];
#pragma unroll
      for (int t = 0; t < 4; ++t) { LOAD_WH(v[2 * t], t, sub, kr, 0); LOAD_WH(v[2 * t + 1], t, sub, kr, 1); }
      PUSH_SETUP();
#pragma unroll
      for (int t = 0; t < 4; ++t) {
        STORE_WH(v[2 * t], (u32)t * W_SLOT + (u32)(kr * 2 + sub) * 8192u);
        STORE_WH(v[2 * t + 1], (u32)t * W_SLOT + (u32)(kr * 2 + sub) * 8192u + 4096u);
      }
      drain();
      if (lane < 8) st_sys(dflag + (WF_OFF >> 2) + (lane >> 1) * 256 + (kr * 2 + sub) * 2 + (lane & 1), call);
      STAMP(1);
    } else {
      int t0 = 0, t1 = 1;
      if (L != 7) {
        while (par(fmask(t0) & L)) ++t0;
        t1 = t0 + 1; while (par(fmask(t1) & L)) ++t1;
      }
      const int s0 = t0 * 4 + ker_index(t0, L), s1 = t1 * 4 + ker_index(t1, L);
      const int xe_rt = 5 + pair_of(L);
      const u32x4 xs = {(u32)(size_t)p.x, (u32)((size_t)p.x >> 32) & 0xffffu, SHARD_BYTES, 0x00020000u};
      const u32x4 wsr = {(u32)(size_t)p.w, (u32)((size_t)p.w >> 32) & 0xffffu, SHARD_BYTES, 0x00020000u};
      u32x4 pa[4], pb[4];
      u32 off_a, fl_a, off_b, fl_b; bool isw_a, isw_b;
#define ALD1(dst, vo, imm) asm volatile("buffer_load_dword %0, %1, %2, 0 offen offset:" #imm : "=v"(dst) : "v"(vo), "s"(wsr) : "memory")
#define ALD4(dst, vo, imm) asm volatile("buffer_load_dwordx4 %0, %1, %2, 0 offen offset:" #imm : "=v"(dst) : "v"(vo), "s"(xs) : "memory")
#define ALD_W8(P, q, vo)                                                       \
          ALD1(P[q][0], vo, 0); ALD1(P[q][1], vo, 512); ALD1(P[q][2], vo, 1024); ALD1(P[q][3], vo, 1536); \
          ALD1(P[q + 1][0], vo, 2048); ALD1(P[q + 1][1], vo, 2560); ALD1(P[q + 1][2], vo, 3072); ALD1(P[q + 1][3], vo, 3584);
#define ALD_X(P, vo) ALD4(P[0], vo, 0); ALD4(P[1], vo, 64); ALD4(P[2], vo, 128); ALD4(P[3], vo, 192);
#define LOAD_PIECE(P, h_, off_, fl_, isw_)                                     \
      {                                                                        \
        const int hk = (h_) / 10, hj = (h_) - 10 * hk;                         \
        if (L == 7 || hj >= 8) {                                               \
          isw_ = false;                                                        \
          const int rt_ = L == 7 ? (hj >> 1) : xe_rt;                          \
          const u32 vo = x_lane + (u32)rt_ * X_RT + (u32)(hj & 1) * 65536u + (u32)hk * 256u; \
          ALD_X(P, vo);                                                        \
          off_ = (L == 7 ? XP_OFF + (u32)rt_ * X_RT : XE_OFF + (u32)(L - 1) * X_RT) + (u32)hk * 8192u + (u32)(hj & 1) * 4096u; \
          fl_ = (L == 7 ? (XPF_OFF >> 2) + (u32)rt_ * 32u : (XEF_OFF >> 2) + (u32)(L - 1) * 32u) + (u32)hk * 2u + (u32)(hj & 1); \
        } else {                                                               \
          const int tt = hj < 4 ? t0 : t1, sf = hj < 4 ? s0 : s1, cb = (hj >> 1) & 1; \
          isw_ = true;                                                         \
          const u32 vo = w_lane + (u32)tt * 128u + (u32)cb * 64u + (u32)(128 * hk + 64 * (hj & 1)) * 512u; \
          const u32 vo2 = vo + 16384u;                                         \
          ALD_W8(P, 0, vo); ALD_W8(P, 2, vo2);                                 \
          off_ = (u32)(sf - tt - 1) * W_SLOT + (u32)(hk * 2 + cb) * 8192u + (u32)(hj & 1) * 4096u; \
          fl_ = (WF_OFF >> 2) + (u32)sf * 64u + (u32)(hk * 2 + cb) * 2u + (u32)(hj & 1); \
        }                                                                      \
      }
      LOAD_PIECE(pa, idx, off_a, fl_a, isw_a);
      LOAD_PIECE(pb, 32 + idx, off_b, fl_b, isw_b);
      PUSH_SETUP();
      u32* lctr = (u32*)(myws + FLAG_OFF + LNK_OFF) + ((call % 3u) * 8u + (u32)L) * 64u;
      bool have_a = true, have_b = true;
      u32 grab = 0;
#define GRAB() if (lane == 0) asm volatile("global_atomic_add %0, %1, %2, off sc0" : "=v"(grab) : "v"(lctr), "v"(1u) : "memory")
      GRAB();
      asm volatile("s_waitcnt vmcnt(1)" ::: "memory");
#define PUSH_STEP(P, off_p, fl_p, isw_p, have_p)                               \
      {                                                                        \
        asm volatile("" : "+v"(P[0]), "+v"(P[1]), "+v"(P[2]), "+v"(P[3]));     \
        if (isw_p) {                                                           \
          _Pragma("unroll") for (int ks = 0; ks < 2; ++ks) {                   \
            u32x4 e, o;                                                        \
            _Pragma("unroll") for (int m = 0; m < 4; ++m) {                    \
              const u32 lo = P[ks * 2 + (m >> 1)][(2 * m) & 3], hi = P[ks * 2 + (m >> 1)][(2 * m + 1) & 3]; \
              e[m] = unpoison(__builtin_amdgcn_perm(hi, lo, 0x05040100u));     \
              o[m] = unpoison(__builtin_amdgcn_perm(hi, lo, 0x07060302u));     \
            }                                                                  \
            __builtin_amdgcn_raw_buffer_store_b128(e, dr, off_p + (u32)lane * 16u + (u32)(ks * 2) * 1024u, 0, AUX_SYS); \
            __builtin_amdgcn_raw_buffer_store_b128(o, dr, off_p + (u32)lane * 16u + (u32)(ks * 2 + 1) * 1024u, 0, AUX_SYS); \
          }                                                                    \
        } else {                                                               \
          _Pragma("unroll") for (int ks = 0; ks < 4; ++ks)                     \
            __builtin_amdgcn_raw_buffer_store_b128(u32x4{unpoison(P[ks][0]), unpoison(P[ks][1]), unpoison(P[ks][2]), unpoison(P[ks][3])}, \
                                                   dr, off_p + (u32)lane * 16u + (u32)ks * 1024u, 0, AUX_SYS); \
        }                                                                      \
        const u32 flc = fl_p;                                                  \
        asm volatile("s_waitcnt vmcnt(4)" : "+v"(grab) :: "memory");           \
        const int h = 64 + (int)__builtin_amdgcn_readfirstlane(grab);          \
        have_p = more && h < 160;                                              \
        more = have_p;                                                         \
        if (have_p) {                                                          \
          GRAB();                                                              \
          LOAD_PIECE(P, h, off_p, fl_p, isw_p);                                \
          if (isw_p) asm volatile("s_waitcnt vmcnt(17)" ::: "memory");         \
          else asm volatile("s_waitcnt vmcnt(5)" ::: "memory");                \
        } else drain();                                                        \
        if (slow && lane == 0) st_sys(dflag + flc, call);                      \
        STAMP(1 + (it < 4 ? it : 4));                                          \
        ++it;                                                                  \
      }
      bool more = true;
      int it = 0;
      for (;;) {
        PUSH_STEP(pa, off_a, fl_a, isw_a, have_a);
        if (!have_b) break;
        PUSH_STEP(pb, off_b, fl_b, isw_b, have_b);
        if (!have_a) break;
      }
      STAMP(6);
    }
    return;
  }

  GET_CALL();
  const u32* myflag = (const u32*)(myws + FLAG_OFF);
  {
    const __amdgpu_buffer_rsrc_t pr = rsrc(myws + ((call + 2u) % 3u) * BUF_BYTES, BUF_BYTES);
    const u32 gid = (u32)c * 256u + threadIdx.x;
#pragma unroll
    for (u32 k = 0; k < (POISON_CHUNKS + 65535u) / 65536u; ++k) {
      const u32 ch = gid + k * 65536u;
      if (ch < POISON_CHUNKS)
        __builtin_amdgcn_raw_buffer_store_b128(u32x4{~0u, ~0u, ~0u, ~0u}, pr, ch * 16u, 0, POISON_AUX);
    }
  }
  if (c < 8 && threadIdx.x == 0)
    __hip_atomic_store((u32*)(myws + FLAG_OFF + LNK_OFF) + (((call + 1u) % 3u) * 8u + (u32)c) * 64u, 0u,
                       __ATOMIC_RELAXED, __HIP_MEMORY_SCOPE_AGENT);
  const int xcd = c & 7, li = c >> 3;
  const int ss = 2 * xcd + (li >> 4);
  const int ti = li & 15;
  const int t = ss >> 2;
  const int s = r ^ ker_elem(t, ss & 3);
  const bool w_own = (ss & 3) == 0;
  const bool x_own = ti < 8;
  int prow, rt;
  u32 x_base, x_lane, x_rb, x_ks, x_kr;
  const u32* xflag = nullptr;
  __amdgpu_buffer_rsrc_t xr;
  if (x_own) {
    prow = r; rt = ti;
    xr = rsrc(p.x, SHARD_BYTES);
    x_base = (u32)rt * X_RT; x_lane = (u32)l16 * 4096u + (u32)g * 16u; x_rb = 65536u; x_ks = 64u; x_kr = 256u;
  } else {
    xr = rsrc(myws + parb, BUF_BYTES);
    x_lane = (u32)lane * 16u; x_rb = 4096u; x_ks = 1024u; x_kr = 8192u;
    if (ti < 13) {
      prow = r ^ 7; rt = ti - 8;
      x_base = XP_OFF + (u32)rt * X_RT;
      xflag = myflag + (XPF_OFF >> 2) + rt * 32;
    } else {
      int e = 0, n = ti - 13;
      for (int q = 1; q <= 6; ++q) if (par(fmask(t) & q)) { if (n == 0) { e = q; } --n; }
      prow = r ^ e; rt = 5 + pair_of(e);
      x_base = XE_OFF + (u32)(e - 1) * X_RT;
      xflag = myflag + (XEF_OFF >> 2) + (e - 1) * 32;
    }
  }
  const u32 w_off = w_own ? OWNW_OFF + (u32)t * W_SLOT : parb + (u32)(ss - t - 1) * W_SLOT;
  const __amdgpu_buffer_rsrc_t wr = rsrc(myws + w_off, W_SLOT);
  const u32* wflag = myflag + (WF_OFF >> 2) + ss * 64;

  const u32* pollp = nullptr;
  if (lane < 8) { if (xflag && slow) pollp = xflag + (4 * (lane >> 1) + wv) * 2 + (lane & 1); }
  else if (lane < 24) { if (slow || w_own) pollp = wflag + (4 * ((lane - 8) >> 2) + wv) * 4 + (lane & 3); }
  const char* probep = nullptr;
  u32 probe_step = 0;
  if (!slow && (lane & PROBE_MASK) == PROBE_MASK) {
    if (lane < 8) { if (!x_own) { probep = myws + parb + x_base + (u32)wv * 8192u + (u32)lane * 1024u + 1020u; probe_step = 32768u; } }
    else if (lane < 24) { if (!w_own) { probep = myws + w_off + (u32)wv * 16384u + (u32)(lane - 8) * 1024u + 1020u; probe_step = 65536u; } }
  }
  const bool use_flags = slow || w_own;
  const bool use_probe = !slow && !(x_own && w_own);

  Waiter wt; wt.tmo = tmo; wt.call = call; wt.t0 = now(); wt.spins = 0; wt.dead = false;
  u32 ready = 0;

  f32x4 acc[2][4];
#pragma unroll
  for (int a = 0; a < 2; ++a)
#pragma unroll
    for (int b = 0; b < 4; ++b) acc[a][b] = f32x4{0.f, 0.f, 0.f, 0.f};
  u32x4 xb[2][8], wb[2][16];

#define WAIT_FOR(mask)                                                        \
  while ((ready & (mask)) != (mask)) {                                        \
    u32 v_ = call;                                                            \
    if (pollp) v_ = ld_sys(pollp);                                            \
    ready = (u32)__builtin_amdgcn_ballot_w64((int)(v_ - call) >= 0);          \
    if ((ready & (mask)) == (mask)) break;                                    \
    if (wt.expired()) { ready = 0xffffffffu; break; }                         \
    __builtin_amdgcn_s_sleep(POLL_SLEEP);                                     \
  }
#define LOADS(step)                                                           \
  {                                                                           \
    const u32 kr_ = 4 * (step) + wv;                                          \
    if (x_own) {                                                              \
      _Pragma("unroll") for (int i = 0; i < 8; ++i)                           \
        xb[(step) & 1][i] = __builtin_amdgcn_raw_buffer_load_b128(            \
            xr, x_base + x_lane + kr_ * x_kr + (u32)(i >> 2) * x_rb + (u32)(i & 3) * x_ks, 0, 0); \
    } else {                                                                  \
      _Pragma("unroll") for (int i = 0; i < 8; ++i)                           \
        xb[(step) & 1][i] = __builtin_amdgcn_raw_buffer_load_b128(            \
            xr, x_base + x_lane + kr_ * x_kr + (u32)(i >> 2) * x_rb + (u32)(i & 3) * x_ks, 0, AUX_SYS); \
    }                                                                         \
    _Pragma("unroll") for (int i = 0; i < 16; ++i)                            \
      wb[(step) & 1][i] = __builtin_amdgcn_raw_buffer_load_b128(              \
          wr, (u32)lane * 16u + kr_ * 16384u + (u32)i * 1024u, 0, AUX_SYS);   \
  }
#define ISSUE(step)                                                           \
  {                                                                           \
    if (use_flags) { WAIT_FOR((0x3u << (2 * (step))) | (0xf00u << (4 * (step)))); } \
    if (use_probe) {                                                          \
      while (__builtin_amdgcn_ballot_w64(pv == 0xffffffffu) != 0) {           \
        if (wt.expired()) break;                                              \
        if ((step) > 0) __builtin_amdgcn_s_sleep(POLL_SLEEP);                 \
        if (probep) pv = ld_sys((const u32*)(probep + (step) * probe_step));  \
      }                                                                       \
    }                                                                         \
    LOADS(step);                                                              \
    if ((step) < 3 && probep) pv = ld_sys((const u32*)(probep + ((step) + 1) * probe_step)); \
  }
  u32 pv = probep ? 0xffffffffu : 0u;
#define VERIFY(step)                                                          \
  if (use_probe) {                                                            \
    for (;;) {                                                                \
      u32 mx_ = 0, mw_ = 0;                                                   \
      _Pragma("unroll") for (int i = 0; i < 8; ++i)                           \
      _Pragma("unroll") for (int q = 0; q < 4; ++q) mx_ = pk_max(mx_, xb[(step) & 1][i][q]); \
      _Pragma("unroll") for (int i = 0; i < 16; ++i)                          \
      _Pragma("unroll") for (int q = 0; q < 4; ++q) mw_ = pk_max(mw_, wb[(step) & 1][i][q]); \
      if (x_own) mx_ = 0;                                                     \
      const u32 m_ = pk_max(mx_, mw_);                                        \
      const bool bad_ = (m_ & 0xffffu) == 0xffffu || (m_ >> 16) == 0xffffu;   \
      if (__builtin_amdgcn_ballot_w64(bad_) == 0) break;                      \
      if (wt.expired()) break;                                                \
      __builtin_amdgcn_s_sleep(POLL_SLEEP);                                   \
      LOADS(step);                                                            \
    }                                                                         \
  }
#define COMPUTE(step)                                                         \
  {                                                                           \
    _Pragma("unroll") for (int ks = 0; ks < 4; ++ks)                          \
    _Pragma("unroll") for (int rb = 0; rb < 2; ++rb)                          \
    _Pragma("unroll") for (int cf = 0; cf < 4; ++cf)                          \
      acc[rb][cf] = __builtin_amdgcn_mfma_f32_16x16x32_bf16(                  \
          __builtin_bit_cast(bf16x8, wb[(step) & 1][(cf >> 1) * 8 + ks * 2 + (cf & 1)]), \
          __builtin_bit_cast(bf16x8, xb[(step) & 1][rb * 4 + ks]), acc[rb][cf], 0, 0, 0); \
  }

  ISSUE(0);
  STAMP(1);
  VERIFY(0);
  ISSUE(1);
  COMPUTE(0);
  STAMP(2);
  VERIFY(1);
  ISSUE(2);
  COMPUTE(1);
  STAMP(3);
  VERIFY(2);
  ISSUE(3);
  COMPUTE(2);
  STAMP(4);
  VERIFY(3);
  COMPUTE(3);
  STAMP(5);

  {
    float* my = red + wv * 32 * LDS_ROW;
#pragma unroll
    for (int rb = 0; rb < 2; ++rb)
#pragma unroll
      for (int cb = 0; cb < 2; ++cb) {
        float* q = my + (16 * rb + l16) * LDS_ROW + 32 * cb + 8 * g;
        const f32x4 e = acc[rb][cb * 2], o = acc[rb][cb * 2 + 1];
        *(f32x4*)q = f32x4{e[0], o[0], e[1], o[1]};
        *(f32x4*)(q + 4) = f32x4{e[2], o[2], e[3], o[3]};
      }
  }
  if (lane == 0) deadf[wv] = wt.dead ? 1u : 0u;
  __syncthreads();
  STAMP(6);
  if (threadIdx.x == 0) __hip_atomic_store(ctr, call, __ATOMIC_RELAXED, __HIP_MEMORY_SCOPE_AGENT);
  const u32 tag = (call & 1u) ? 0x80008000u : 0u;
  const __amdgpu_buffer_rsrc_t myout = rsrc(p.out[r], SHARD_BYTES);
  const int row = 8 * wv + (lane >> 3), col = (lane & 7) * 8;
  {
    const float* q = red + row * LDS_ROW + col;
    f32x4 a = *(const f32x4*)q, b = *(const f32x4*)(q + 4);
#pragma unroll
    for (int w2 = 1; w2 < 4; ++w2) {
      a += *(const f32x4*)(q + w2 * 32 * LDS_ROW);
      b += *(const f32x4*)(q + w2 * 32 * LDS_ROW + 4);
    }
    float y[8] = {a[0], a[1], a[2], a[3], b[0], b[1], b[2], b[3]};
    u32 h[8];
#pragma unroll
    for (int i = 0; i < 8; ++i) {
      const float v = y[i] < 0.f ? 0.f : y[i];
      const u32 u = __builtin_bit_cast(u32, v);
      h[i] = (v != v) ? 0x7fc0u : (((u + 0x7fffu + ((u >> 16) & 1u)) >> 16) & 0x7fffu);
    }
    const u32 anydead = deadf[0] | deadf[1] | deadf[2] | deadf[3];
    u32x4 o;
    if (anydead) o = u32x4{0x7fc07fc0u, 0x7fc07fc0u, 0x7fc07fc0u, 0x7fc07fc0u};
    else o = u32x4{h[0] | (h[1] << 16), h[2] | (h[3] << 16), h[4] | (h[5] << 16), h[6] | (h[7] << 16)};
    if (s == r) {
      __builtin_amdgcn_raw_buffer_store_b128(o, myout, (u32)(256 * prow + 32 * rt + row) * 512u + (u32)(64 * t + col) * 2u, 0, 0);
    } else {
      const __amdgpu_buffer_rsrc_t orr = rsrc(p.ws[s] + OUTS_OFF, 1u << 20);
      __builtin_amdgcn_raw_buffer_store_b128(o | tag, orr, (u32)((8 * prow + rt) * 4 + t) * 4096u + (u32)threadIdx.x * 16u, 0, AUX_SYS);
    }
  }
  STAMP(7);
  {
    const int ft = c & 3, frt = (c >> 2) & 7, fp = c >> 5;
    const int fd = fp ^ r;
    const bool local = par(fmask(ft) & fd) == 0 ? (fd == 0)
                       : frt < 5 ? (fd == 7) : (fd != 7 && pair_of(fd) == frt - 5);
    if (!local) {
      const __amdgpu_buffer_rsrc_t ir = rsrc(myws + OUTS_OFF, 1u << 20);
      u32x4 v;
      for (;;) {
        v = __builtin_amdgcn_raw_buffer_load_b128(ir, (u32)c * 4096u + (u32)threadIdx.x * 16u, 0, AUX_SYS);
        const u32 bad = ((v[0] ^ tag) | (v[1] ^ tag) | (v[2] ^ tag) | (v[3] ^ tag)) & 0x80008000u;
        if (__builtin_amdgcn_ballot_w64(bad != 0u) == 0) { v &= 0x7fff7fffu; break; }
        if (wt.expired()) { v = u32x4{0x7fc07fc0u, 0x7fc07fc0u, 0x7fc07fc0u, 0x7fc07fc0u}; break; }
        __builtin_amdgcn_s_sleep(POLL_SLEEP);
      }
      __builtin_amdgcn_raw_buffer_store_b128(v, myout, (u32)(256 * fp + 32 * frt + row) * 512u + (u32)(64 * ft + col) * 2u, 0, 0);
    }
  }
  STAMP(8);
}

}

extern "C" void kernel_launch(int rank, int nranks, ncclComm_t comm,
                              hipStream_t stream,
                              void* const* d_in, const int* in_sizes, int n_in,
                              void* d_out, int out_size,
                              const int* in_dtypes, int out_dtype,
                              void* const* d_out_all,
                              void* const* d_ws_all, size_t ws_bytes) {
  (void)comm; (void)in_dtypes; (void)out_dtype; (void)d_out;
  if (nranks != NR || n_in != 2 || in_sizes[0] != 256 * 2048 || in_sizes[1] != 2048 * 256 ||
      out_size != 2048 * 256 || ws_bytes < (size_t)FLAG_OFF + FLAG_BYTES) {
    fprintf(stderr, "kernel_launch: unexpected problem shape\n");
    return;
  }
  Params p;
  for (int i = 0; i < NR; ++i) { p.ws[i] = (char*)d_ws_all[i]; p.out[i] = (char*)d_out_all[i]; }
  p.x = (const char*)d_in[0];
  p.w = (const char*)d_in[1];
  p.rank = rank;
  ag_gemm_kernel<<<256, 320, 0, stream>>>(p);
  HIP_CHECK(hipGetLastError());
}
```

```cpp
#include <hip/hip_runtime.h>
#include <rccl/rccl.h>
#include <cstdio>

#define HIP_CHECK(expr) do { hipError_t _e = (expr); if (_e != hipSuccess) { \
    fprintf(stderr, "HIP error %s at %s:%d\n", hipGetErrorString(_e), __FILE__, __LINE__); \
    return; } } while (0)

namespace {

typedef unsigned u32;
typedef unsigned long long u64;
typedef u32 u32x4 __attribute__((ext_vector_type(4)));
typedef __bf16 bf16x8 __attribute__((ext_vector_type(8)));
typedef float f32x4 __attribute__((ext_vector_type(4)));
typedef unsigned short u16x2 __attribute__((ext_vector_type(2)));

constexpr int NR = 8;
constexpr u32 KIB = 1024u;
constexpr u32 W_SLOT = 256u * KIB;
constexpr u32 X_RT = 128u * KIB;
constexpr u32 XP_OFF = 12u * W_SLOT;
constexpr u32 XE_OFF = XP_OFF + 5u * X_RT;
constexpr u32 BUF_BYTES = XE_OFF + 6u * X_RT;
constexpr u32 OWNW_OFF = 3u * BUF_BYTES;
constexpr u32 OUTS_OFF = OWNW_OFF + 4u * W_SLOT;
constexpr u32 POISON_CHUNKS = BUF_BYTES / 16u;
constexpr u32 FLAG_OFF = 16u << 20;
constexpr u32 CTR_OFF = 0;
constexpr u32 TMO_OFF = 32u * KIB;
constexpr u32 LNK_OFF = 48u * KIB;
constexpr u32 WF_OFF = 36u * KIB;
constexpr u32 XPF_OFF = 40u * KIB;
constexpr u32 XEF_OFF = 44u * KIB;
constexpr u32 DBG_OFF = 256u * KIB;
constexpr u32 FLAG_BYTES = 1u << 20;
constexpr u32 SHARD_BYTES = 1u << 20;
constexpr u64 DEADLINE = 100000000ull;
constexpr int AUX_SYS = 17;
#define POLL_WIN 4
#define BULK_AUX 17
#define POISON_EARLY 0
#define POISON_AUX 17
#define PACE 2
#define PROBE_MASK 3
#define POLL_SLEEP 1
constexpr int LDS_ROW = 68;

struct Params {
  char* myws;
  char* myout;
  char* ws[NR];
  char* out[NR];
  const char* x;
  const char* w;
  int rank;
};

__device__ __forceinline__ u64 now() { return __builtin_amdgcn_s_memrealtime(); }
__device__ __forceinline__ int par(int v) { return __builtin_popcount(v) & 1; }
__device__ __forceinline__ int fmask(int t) { return t == 3 ? 7 : (1 << t); }
__device__ __forceinline__ int ker_index(int t, int d) {
  return t == 0 ? (d >> 1) : t == 1 ? ((d & 1) | ((d >> 2) << 1)) : t == 2 ? (d & 3) : (d >> 1);
}
__device__ __forceinline__ int ker_elem(int t, int j) {
  return t == 0 ? (j << 1) : t == 1 ? ((j & 1) | ((j >> 1) << 2)) : t == 2 ? j : ((j << 1) | par(j));
}
__device__ __forceinline__ int pair_of(int d) { return (d == 1 || d == 6) ? 0 : (d == 2 || d == 5) ? 1 : 2; }
__device__ __forceinline__ __amdgpu_buffer_rsrc_t rsrc(const void* p, u32 bytes) {
  return __builtin_amdgcn_make_buffer_rsrc(const_cast<void*>(p), 0, (int)bytes, 0x00020000);
}
__device__ __forceinline__ u32 ld_sys(const u32* p) {
  return __hip_atomic_load(p, __ATOMIC_RELAXED, __HIP_MEMORY_SCOPE_SYSTEM);
}
__device__ __forceinline__ void st_sys(u32* p, u32 v) {
  __hip_atomic_store(p, v, __ATOMIC_RELAXED, __HIP_MEMORY_SCOPE_SYSTEM);
}
__device__ __forceinline__ void drain() { asm volatile("s_waitcnt vmcnt(0)" ::: "memory"); }
__device__ __forceinline__ u32 pk_min(u32 a, u32 b) {
  return __builtin_bit_cast(u32, __builtin_elementwise_min(__builtin_bit_cast(u16x2, a), __builtin_bit_cast(u16x2, b)));
}
__device__ __forceinline__ u32 pk_max(u32 a, u32 b) {
  return __builtin_bit_cast(u32, __builtin_elementwise_max(__builtin_bit_cast(u16x2, a), __builtin_bit_cast(u16x2, b)));
}
__device__ __forceinline__ u32 unpoison(u32 v) { return pk_min(v, 0xfffefffeu); }

#define STAMP(i) do { } while (0)

struct Waiter {
  u32* tmo;
  u32 call;
  u64 t0;
  u32 spins;
  bool dead;
  __device__ __forceinline__ bool expired() {
    if (dead) return true;
    ++spins;
    if ((spins & 15u) == 0u) {
      u32 t = __hip_atomic_load(tmo, __ATOMIC_RELAXED, __HIP_MEMORY_SCOPE_AGENT);
      t = __builtin_amdgcn_readfirstlane(t);
      if (t == call) dead = true;
      else if (now() - t0 > DEADLINE) {
        __hip_atomic_store(tmo, call, __ATOMIC_RELAXED, __HIP_MEMORY_SCOPE_AGENT);
        dead = true;
      }
    }
    return dead;
  }
};

__global__ void __launch_bounds__(320) ag_gemm_kernel(Params p) {
  __shared__ __attribute__((aligned(16))) float red[4 * 32 * LDS_ROW];
  __shared__ u32 deadf[4];
  const int lane = threadIdx.x & 63;
  const int wv = __builtin_amdgcn_readfirstlane(threadIdx.x >> 6);
  const int c = blockIdx.x;
  const int r = p.rank;
  char* myws = p.myws;
  u32* ctr = (u32*)(myws + FLAG_OFF + CTR_OFF + (u32)c * 128u);
  u32* tmo = (u32*)(myws + FLAG_OFF + TMO_OFF);
  STAMP(0);
  const u32 ctrv = __hip_atomic_load(ctr, __ATOMIC_RELAXED, __HIP_MEMORY_SCOPE_AGENT);
  const int g = lane >> 4, l16 = lane & 15;
#define GET_CALL()                                                             \
  const u32 call = __builtin_amdgcn_readfirstlane(ctrv) + 1u;                  \
  const u32 parb = (call % 3u) * BUF_BYTES;                                    \
  const bool slow = call <= 2u;

#define POISON()                                                               \
  {                                                                            \
    const __amdgpu_buffer_rsrc_t pr = rsrc(myws + ((call + 2u) % 3u) * BUF_BYTES, BUF_BYTES); \
    const u32 per = POISON_EARLY ? POISON_CHUNKS / 1024u : POISON_CHUNKS / 256u; \
    const u32 base = (POISON_EARLY ? (u32)(c * 4 + wv) : (u32)c) * per;        \
    _Pragma("unroll") for (u32 k = 0; k < (per + 63u) / 64u; ++k) {            \
      const u32 ch = k * 64u + (u32)lane;                                      \
      if (ch < per)                                                            \
        __builtin_amdgcn_raw_buffer_store_b128(u32x4{~0u, ~0u, ~0u, ~0u}, pr, (base + ch) * 16u, 0, POISON_AUX); \
    }                                                                          \
  }

  if (wv == 4) {
    const int L = (c + (c >> 3)) & 7;
    const int idx = c >> 3;
    const int sub = idx & 1, kr = idx >> 1;
    const int dst = r ^ L;
    char* dws = p.ws[dst];
    u32* dflag = (u32*)(dws + FLAG_OFF);
    const __amdgpu_buffer_rsrc_t xr = rsrc(p.x, SHARD_BYTES);
    const __amdgpu_buffer_rsrc_t wr = rsrc(p.w, SHARD_BYTES);
    __builtin_amdgcn_s_setprio(3);
    const u32 w_lane = (u32)(8 * g) * 512u + (u32)(2 * l16) * 2u;
    const u32 x_lane = (u32)l16 * 4096u + (u32)(8 * g) * 2u;
#define LOAD_WH(v, t, cb, kr_, hf)                                             \
    _Pragma("unroll") for (int i = 0; i < 16; ++i)                             \
      v[i] = __builtin_amdgcn_raw_buffer_load_b32(wr, w_lane + (u32)(t) * 128u + (u32)(cb) * 64u + \
                 (u32)(128 * (kr_) + 64 * (hf) + (i >> 3) * 32 + (i & 7)) * 512u, 0, 0);
#define STORE_WH(v, off)                                                       \
    _Pragma("unroll") for (int ks = 0; ks < 2; ++ks) {                         \
      u32x4 e, o;                                                              \
      _Pragma("unroll") for (int m = 0; m < 4; ++m) {                          \
        e[m] = unpoison(__builtin_amdgcn_perm(v[ks * 8 + 2 * m + 1], v[ks * 8 + 2 * m], 0x05040100u)); \
        o[m] = unpoison(__builtin_amdgcn_perm(v[ks * 8 + 2 * m + 1], v[ks * 8 + 2 * m], 0x07060302u)); \
      }                                                                        \
      __builtin_amdgcn_raw_buffer_store_b128(e, dr, (u32)(off) + (u32)lane * 16u + (u32)(ks * 2) * 1024u, 0, AUX_SYS); \
      __builtin_amdgcn_raw_buffer_store_b128(o, dr, (u32)(off) + (u32)lane * 16u + (u32)(ks * 2 + 1) * 1024u, 0, AUX_SYS); \
    }
#define LOAD_XH(v, rt, rb, kr_)                                                \
    _Pragma("unroll") for (int ks = 0; ks < 4; ++ks) {                         \
      const u32x4 q_ = __builtin_amdgcn_raw_buffer_load_b128(xr, x_lane + (u32)(rt) * X_RT + (u32)(rb) * 65536u + (u32)(kr_) * 256u + (u32)ks * 64u, 0, 0); \
      v[ks * 4] = q_[0]; v[ks * 4 + 1] = q_[1]; v[ks * 4 + 2] = q_[2]; v[ks * 4 + 3] = q_[3]; \
    }
#define STORE_XH(v, off)                                                       \
    _Pragma("unroll") for (int ks = 0; ks < 4; ++ks)                           \
      __builtin_amdgcn_raw_buffer_store_b128(u32x4{unpoison(v[ks * 4]), unpoison(v[ks * 4 + 1]), unpoison(v[ks * 4 + 2]), unpoison(v[ks * 4 + 3])}, dr, (u32)(off) + (u32)lane * 16u + (u32)ks * 1024u, 0, AUX_SYS);
#define PUSH_SETUP()                                                           \
    __builtin_amdgcn_sched_barrier(0);                                         \
    GET_CALL();                                                                \
    STAMP(10);                                                                 \
    const __amdgpu_buffer_rsrc_t dr = rsrc(dws + (L == 0 ? OWNW_OFF : parb), BUF_BYTES);
#define WAITCNT_(n) asm volatile("s_waitcnt vmcnt(" #n ")" ::: "memory")
#define WAITCNT(n) do { if ((n) == 0) WAITCNT_(0); else if ((n) == 4) WAITCNT_(4); else if ((n) == 8) WAITCNT_(8); else WAITCNT_(12); } while (0)
    if (L == 0) {
      u32 v[8][16];
#pragma unroll
      for (int t = 0; t < 4; ++t) { LOAD_WH(v[2 * t], t, sub, kr, 0); LOAD_WH(v[2 * t + 1], t, sub, kr, 1); }
      PUSH_SETUP();
#pragma unroll
      for (int t = 0; t < 4; ++t) {
        STORE_WH(v[2 * t], (u32)t * W_SLOT + (u32)(kr * 2 + sub) * 8192u);
        STORE_WH(v[2 * t + 1], (u32)t * W_SLOT + (u32)(kr * 2 + sub) * 8192u + 4096u);
      }
      drain();
      if (lane < 8) st_sys(dflag + (WF_OFF >> 2) + (lane >> 1) * 256 + (kr * 2 + sub) * 2 + (lane & 1), call);
      STAMP(1);
      POISON();
    } else {
      int t0 = 0, t1 = 1;
      if (L != 7) {
        while (par(fmask(t0) & L)) ++t0;
        t1 = t0 + 1; while (par(fmask(t1) & L)) ++t1;
      }
      const int s0 = t0 * 4 + ker_index(t0, L), s1 = t1 * 4 + ker_index(t1, L);
      const int xe_rt = 5 + pair_of(L);
      const u32x4 xs = {(u32)(size_t)p.x, (u32)((size_t)p.x >> 32) & 0xffffu, SHARD_BYTES, 0x00020000u};
      const u32x4 wsr = {(u32)(size_t)p.w, (u32)((size_t)p.w >> 32) & 0xffffu, SHARD_BYTES, 0x00020000u};
      u32x4 pa[4], pb[4];
      u32 off_a, fl_a, off_b, fl_b; bool isw_a, isw_b;
#define ALD1(dst, vo, imm) asm volatile("buffer_load_dword %0, %1, %2, 0 offen offset:" #imm : "=v"(dst) : "v"(vo), "s"(wsr) : "memory")
#define ALD4(dst, vo, imm) asm volatile("buffer_load_dwordx4 %0, %1, %2, 0 offen offset:" #imm : "=v"(dst) : "v"(vo), "s"(xs) : "memory")
#define ALD_W8(P, q, vo)                                                       \
          ALD1(P[q][0], vo, 0); ALD1(P[q][1], vo, 512); ALD1(P[q][2], vo, 1024); ALD1(P[q][3], vo, 1536); \
          ALD1(P[q + 1][0], vo, 2048); ALD1(P[q + 1][1], vo, 2560); ALD1(P[q + 1][2], vo, 3072); ALD1(P[q + 1][3], vo, 3584);
#define ALD_X(P, vo) ALD4(P[0], vo, 0); ALD4(P[1], vo, 64); ALD4(P[2], vo, 128); ALD4(P[3], vo, 192);
#define LOAD_PIECE(P, h_, off_, fl_, isw_)                                     \
      {                                                                        \
        const int hk = (h_) / 10, hj = (h_) - 10 * hk;                         \
        if (L == 7 || hj >= 8) {                                               \
          isw_ = false;                                                        \
          const int rt_ = L == 7 ? (hj >> 1) : xe_rt;                          \
          const u32 vo = x_lane + (u32)rt_ * X_RT + (u32)(hj & 1) * 65536u + (u32)hk * 256u; \
          ALD_X(P, vo);                                                        \
          off_ = (L == 7 ? XP_OFF + (u32)rt_ * X_RT : XE_OFF + (u32)(L - 1) * X_RT) + (u32)hk * 8192u + (u32)(hj & 1) * 4096u; \
          fl_ = (L == 7 ? (XPF_OFF >> 2) + (u32)rt_ * 32u : (XEF_OFF >> 2) + (u32)(L - 1) * 32u) + (u32)hk * 2u + (u32)(hj & 1); \
        } else {                                                               \
          const int tt = hj < 4 ? t0 : t1, sf = hj < 4 ? s0 : s1, cb = (hj >> 1) & 1; \
          isw_ = true;                                                         \
          const u32 vo = w_lane + (u32)tt * 128u + (u32)cb * 64u + (u32)(128 * hk + 64 * (hj & 1)) * 512u; \
          const u32 vo2 = vo + 16384u;                                         \
          ALD_W8(P, 0, vo); ALD_W8(P, 2, vo2);                                 \
          off_ = (u32)(sf - tt - 1) * W_SLOT + (u32)(hk * 2 + cb) * 8192u + (u32)(hj & 1) * 4096u; \
          fl_ = (WF_OFF >> 2) + (u32)sf * 64u + (u32)(hk * 2 + cb) * 2u + (u32)(hj & 1); \
        }                                                                      \
      }
      LOAD_PIECE(pa, idx, off_a, fl_a, isw_a);
      LOAD_PIECE(pb, 32 + idx, off_b, fl_b, isw_b);
      PUSH_SETUP();
      u32* lctr = (u32*)(myws + FLAG_OFF + LNK_OFF) + ((call % 3u) * 8u + (u32)L) * 64u;
      bool have_a = true, have_b = true;
      u32 grab = 0;
#define GRAB() if (lane == 0) asm volatile("global_atomic_add %0, %1, %2, off sc0" : "=v"(grab) : "v"(lctr), "v"(1u) : "memory")
      GRAB();
      asm volatile("s_waitcnt vmcnt(1)" ::: "memory");
#define PUSH_STEP(P, off_p, fl_p, isw_p, have_p)                               \
      {                                                                        \
        asm volatile("" : "+v"(P[0]), "+v"(P[1]), "+v"(P[2]), "+v"(P[3]));     \
        if (isw_p) {                                                           \
          _Pragma("unroll") for (int ks = 0; ks < 2; ++ks) {                   \
            u32x4 e, o;                                                        \
            _Pragma("unroll") for (int m = 0; m < 4; ++m) {                    \
              const u32 lo = P[ks * 2 + (m >> 1)][(2 * m) & 3], hi = P[ks * 2 + (m >> 1)][(2 * m + 1) & 3]; \
              e[m] = unpoison(__builtin_amdgcn_perm(hi, lo, 0x05040100u));     \
              o[m] = unpoison(__builtin_amdgcn_perm(hi, lo, 0x07060302u));     \
            }                                                                  \
            __builtin_amdgcn_raw_buffer_store_b128(e, dr, off_p + (u32)lane * 16u + (u32)(ks * 2) * 1024u, 0, AUX_SYS); \
            __builtin_amdgcn_raw_buffer_store_b128(o, dr, off_p + (u32)lane * 16u + (u32)(ks * 2 + 1) * 1024u, 0, AUX_SYS); \
          }                                                                    \
        } else {                                                               \
          _Pragma("unroll") for (int ks = 0; ks < 4; ++ks)                     \
            __builtin_amdgcn_raw_buffer_store_b128(u32x4{unpoison(P[ks][0]), unpoison(P[ks][1]), unpoison(P[ks][2]), unpoison(P[ks][3])}, \
                                                   dr, off_p + (u32)lane * 16u + (u32)ks * 1024u, 0, AUX_SYS); \
        }                                                                      \
        const u32 flc = fl_p;                                                  \
        asm volatile("s_waitcnt vmcnt(4)" : "+v"(grab) :: "memory");           \
        const int h = 64 + (int)__builtin_amdgcn_readfirstlane(grab);          \
        have_p = more && h < 160;                                              \
        more = have_p;                                                         \
        if (have_p) {                                                          \
          GRAB();                                                              \
          LOAD_PIECE(P, h, off_p, fl_p, isw_p);                                \
          if (isw_p) asm volatile("s_waitcnt vmcnt(17)" ::: "memory");         \
          else asm volatile("s_waitcnt vmcnt(5)" ::: "memory");                \
        } else drain();                                                        \
        if (slow && lane == 0) st_sys(dflag + flc, call);                      \
        STAMP(1 + (it < 4 ? it : 4));                                          \
        ++it;                                                                  \
      }
      bool more = true;
      int it = 0;
      for (;;) {
        PUSH_STEP(pa, off_a, fl_a, isw_a, have_a);
        if (!have_b) break;
        PUSH_STEP(pb, off_b, fl_b, isw_b, have_b);
        if (!have_a) break;
      }
      STAMP(6);
      POISON();
    }
    return;
  }

  GET_CALL();
  const u32* myflag = (const u32*)(myws + FLAG_OFF);
  if (c < 8 && threadIdx.x == 0)
    __hip_atomic_store((u32*)(myws + FLAG_OFF + LNK_OFF) + (((call + 1u) % 3u) * 8u + (u32)c) * 64u, 0u,
                       __ATOMIC_RELAXED, __HIP_MEMORY_SCOPE_AGENT);
  const int xcd = c & 7, li = c >> 3;
  const int ss = 2 * xcd + (li >> 4);
  const int ti = li & 15;
  const int t = ss >> 2;
  const int s = r ^ ker_elem(t, ss & 3);
  const bool w_own = (ss & 3) == 0;
  const bool x_own = ti < 8;
  int prow, rt;
  u32 x_base, x_lane, x_rb, x_ks, x_kr;
  const u32* xflag = nullptr;
  __amdgpu_buffer_rsrc_t xr;
  if (x_own) {
    prow = r; rt = ti;
    xr = rsrc(p.x, SHARD_BYTES);
    x_base = (u32)rt * X_RT; x_lane = (u32)l16 * 4096u + (u32)g * 16u; x_rb = 65536u; x_ks = 64u; x_kr = 256u;
  } else {
    xr = rsrc(myws + parb, BUF_BYTES);
    x_lane = (u32)lane * 16u; x_rb = 4096u; x_ks = 1024u; x_kr = 8192u;
    if (ti < 13) {
      prow = r ^ 7; rt = ti - 8;
      x_base = XP_OFF + (u32)rt * X_RT;
      xflag = myflag + (XPF_OFF >> 2) + rt * 32;
    } else {
      int e = 0, n = ti - 13;
      for (int q = 1; q <= 6; ++q) if (par(fmask(t) & q)) { if (n == 0) { e = q; } --n; }
      prow = r ^ e; rt = 5 + pair_of(e);
      x_base = XE_OFF + (u32)(e - 1) * X_RT;
      xflag = myflag + (XEF_OFF >> 2) + (e - 1) * 32;
    }
  }
  const u32 w_off = w_own ? OWNW_OFF + (u32)t * W_SLOT : parb + (u32)(ss - t - 1) * W_SLOT;
  const __amdgpu_buffer_rsrc_t wr = rsrc(myws + w_off, W_SLOT);
  const u32* wflag = myflag + (WF_OFF >> 2) + ss * 64;

  const u32* pollp = nullptr;
  bool poll_flag = false;
  {
    const int pst = lane >> 3, pc = lane & 7, pkr = 4 * pst + wv;
    if (lane < 32 && pc < 2) {
      if (!x_own) {
        poll_flag = slow;
        pollp = slow ? xflag + pkr * 2 + pc
                     : (const u32*)(myws + parb + x_base + (u32)pkr * 8192u + (u32)pc * 4096u + 4092u);
      }
    } else if (lane < 32 && pc < 6) {
      poll_flag = slow || w_own;
      pollp = poll_flag ? wflag + pkr * 4 + (pc - 2)
                        : (const u32*)(myws + w_off + (u32)pkr * 16384u + (u32)(pc - 2) * 4096u + 4092u);
    }
  }
  const bool use_probe = true;

  Waiter wt; wt.tmo = tmo; wt.call = call; wt.t0 = now(); wt.spins = 0; wt.dead = false;
  u32 done = 0;

  f32x4 acc[2][4];
#pragma unroll
  for (int a = 0; a < 2; ++a)
#pragma unroll
    for (int b = 0; b < 4; ++b) acc[a][b] = f32x4{0.f, 0.f, 0.f, 0.f};
  u32x4 xb[2][8], wb[2][16];

#define LOADS(buf, kr_, AUX)                                                  \
  {                                                                           \
    if (x_own) {                                                              \
      _Pragma("unroll") for (int i = 0; i < 8; ++i)                           \
        xb[buf][i] = __builtin_amdgcn_raw_buffer_load_b128(                   \
            xr, x_base + x_lane + (kr_) * x_kr + (u32)(i >> 2) * x_rb + (u32)(i & 3) * x_ks, 0, 0); \
    } else {                                                                  \
      _Pragma("unroll") for (int i = 0; i < 8; ++i)                           \
        xb[buf][i] = __builtin_amdgcn_raw_buffer_load_b128(                   \
            xr, x_base + x_lane + (kr_) * x_kr + (u32)(i >> 2) * x_rb + (u32)(i & 3) * x_ks, 0, AUX); \
    }                                                                         \
    _Pragma("unroll") for (int i = 0; i < 16; ++i)                            \
      wb[buf][i] = __builtin_amdgcn_raw_buffer_load_b128(                     \
          wr, (u32)lane * 16u + (kr_) * 16384u + (u32)i * 1024u, 0, AUX);     \
  }
#define DBG_POLL(first)
#define POLL() { const int lo_ = __builtin_ctz(~done); \
    pv = (lane >> 3) >= lo_ + POLL_WIN ? (poll_flag ? call - 1u : 0xffffffffu) : pollp ? ld_sys(pollp) : (poll_flag ? call : 0u); }
#define ISSUE(buf, first)                                                     \
  {                                                                           \
    u32 avail;                                                                \
    for (;;) {                                                                \
      if (first) POLL();                                                      \
      const bool ok_ = poll_flag ? ((int)(pv - call) >= 0) : (pv != 0xffffffffu); \
      const u32 m_ = (u32)__builtin_amdgcn_ballot_w64(ok_);                   \
      DBG_POLL(first);                                                        \
      avail = 0;                                                              \
      _Pragma("unroll") for (int q = 0; q < 4; ++q)                           \
        if (((m_ >> (8 * q)) & 0x3fu) == 0x3fu) avail |= 1u << q;             \
      avail &= ~done;                                                         \
      if (avail) break;                                                       \
      if (wt.expired()) { avail = ~done & 0xfu; break; }                      \
      __builtin_amdgcn_s_sleep(POLL_SLEEP);                                   \
      if (!(first)) POLL();                                                   \
    }                                                                         \
    const u32 st_ = (u32)__builtin_ctz(avail);                                \
    done |= 1u << st_;                                                        \
    krb[buf] = 4u * st_ + (u32)wv;                                            \
    LOADS(buf, krb[buf], BULK_AUX);                                           \
    if (done != 0xfu) POLL();                                                 \
  }
  u32 pv = 0, krb[2];
#define VERIFY(buf)                                                           \
  if (use_probe) {                                                            \
    for (;;) {                                                                \
      u32 mx_ = 0, mw_ = 0;                                                   \
      _Pragma("unroll") for (int i = 0; i < 8; ++i)                           \
      _Pragma("unroll") for (int q = 0; q < 4; ++q) mx_ = pk_max(mx_, xb[buf][i][q]); \
      _Pragma("unroll") for (int i = 0; i < 16; ++i)                          \
      _Pragma("unroll") for (int q = 0; q < 4; ++q) mw_ = pk_max(mw_, wb[buf][i][q]); \
      if (x_own) mx_ = 0;                                                     \
      const u32 m_ = pk_max(mx_, mw_);                                        \
      const bool bad_ = (m_ & 0xffffu) == 0xffffu || (m_ >> 16) == 0xffffu;   \
      if (__builtin_amdgcn_ballot_w64(bad_) == 0) break;                      \
      if (wt.expired()) break;                                                \
      __builtin_amdgcn_s_sleep(POLL_SLEEP);                                   \
      LOADS(buf, krb[buf], AUX_SYS);                                          \
    }                                                                         \
  }
#define COMPUTE(buf)                                                          \
  {                                                                           \
    _Pragma("unroll") for (int ks = 0; ks < 4; ++ks)                          \
    _Pragma("unroll") for (int rb = 0; rb < 2; ++rb)                          \
    _Pragma("unroll") for (int cf = 0; cf < 4; ++cf)                          \
      acc[rb][cf] = __builtin_amdgcn_mfma_f32_16x16x32_bf16(                  \
          __builtin_bit_cast(bf16x8, wb[buf][(cf >> 1) * 8 + ks * 2 + (cf & 1)]), \
          __builtin_bit_cast(bf16x8, xb[buf][rb * 4 + ks]), acc[rb][cf], 0, 0, 0); \
  }

  ISSUE(0, true);
  STAMP(1);
  VERIFY(0);
  ISSUE(1, false);
  COMPUTE(0);
  STAMP(2);
  VERIFY(1);
  ISSUE(0, false);
  COMPUTE(1);
  STAMP(3);
  VERIFY(0);
  ISSUE(1, false);
  COMPUTE(0);
  STAMP(4);
  VERIFY(1);
  COMPUTE(1);
  STAMP(5);

  {
    float* my = red + wv * 32 * LDS_ROW;
#pragma unroll
    for (int rb = 0; rb < 2; ++rb)
#pragma unroll
      for (int cb = 0; cb < 2; ++cb) {
        float* q = my + (16 * rb + l16) * LDS_ROW + 32 * cb + 8 * g;
        const f32x4 e = acc[rb][cb * 2], o = acc[rb][cb * 2 + 1];
        *(f32x4*)q = f32x4{e[0], o[0], e[1], o[1]};
        *(f32x4*)(q + 4) = f32x4{e[2], o[2], e[3], o[3]};
      }
  }
  if (lane == 0) deadf[wv] = wt.dead ? 1u : 0u;
  __syncthreads();
  STAMP(6);
  if (threadIdx.x == 0) __hip_atomic_store(ctr, call, __ATOMIC_RELAXED, __HIP_MEMORY_SCOPE_AGENT);
  const u32 tag = (call & 1u) ? 0x80008000u : 0u;
  const __amdgpu_buffer_rsrc_t myout = rsrc(p.myout, SHARD_BYTES);
  const int row = 8 * wv + (lane >> 3), col = (lane & 7) * 8;
  {
    const float* q = red + row * LDS_ROW + col;
    f32x4 a = *(const f32x4*)q, b = *(const f32x4*)(q + 4);
#pragma unroll
    for (int w2 = 1; w2 < 4; ++w2) {
      a += *(const f32x4*)(q + w2 * 32 * LDS_ROW);
      b += *(const f32x4*)(q + w2 * 32 * LDS_ROW + 4);
    }
    float y[8] = {a[0], a[1], a[2], a[3], b[0], b[1], b[2], b[3]};
    u32 h[8];
#pragma unroll
    for (int i = 0; i < 8; ++i) {
      const float v = y[i] < 0.f ? 0.f : y[i];
      const u32 u = __builtin_bit_cast(u32, v);
      h[i] = (v != v) ? 0x7fc0u : (((u + 0x7fffu + ((u >> 16) & 1u)) >> 16) & 0x7fffu);
    }
    const u32 anydead = deadf[0] | deadf[1] | deadf[2] | deadf[3];
    u32x4 o;
    if (anydead) o = u32x4{0x7fc07fc0u, 0x7fc07fc0u, 0x7fc07fc0u, 0x7fc07fc0u};
    else o = u32x4{h[0] | (h[1] << 16), h[2] | (h[3] << 16), h[4] | (h[5] << 16), h[6] | (h[7] << 16)};
    if (s == r) {
      __builtin_amdgcn_raw_buffer_store_b128(o, myout, (u32)(256 * prow + 32 * rt + row) * 512u + (u32)(64 * t + col) * 2u, 0, 0);
    } else {
      const __amdgpu_buffer_rsrc_t orr = rsrc(p.ws[s] + OUTS_OFF, 1u << 20);
      __builtin_amdgcn_raw_buffer_store_b128(o | tag, orr, (u32)((8 * prow + rt) * 4 + t) * 4096u + (u32)threadIdx.x * 16u, 0, AUX_SYS);
    }
  }
  STAMP(7);
  {
    const int ft = c & 3, frt = (c >> 2) & 7, fp = c >> 5;
    const int fd = fp ^ r;
    const bool local = par(fmask(ft) & fd) == 0 ? (fd == 0)
                       : frt < 5 ? (fd == 7) : (fd != 7 && pair_of(fd) == frt - 5);
    if (!local) {
      const __amdgpu_buffer_rsrc_t ir = rsrc(myws + OUTS_OFF, 1u << 20);
      u32x4 v;
      for (;;) {
        v = __builtin_amdgcn_raw_buffer_load_b128(ir, (u32)c * 4096u + (u32)threadIdx.x * 16u, 0, AUX_SYS);
        const u32 bad = ((v[0] ^ tag) | (v[1] ^ tag) | (v[2] ^ tag) | (v[3] ^ tag)) & 0x80008000u;
        if (__builtin_amdgcn_ballot_w64(bad != 0u) == 0) { v &= 0x7fff7fffu; break; }
        if (wt.expired()) { v = u32x4{0x7fc07fc0u, 0x7fc07fc0u, 0x7fc07fc0u, 0x7fc07fc0u}; break; }
        __builtin_amdgcn_s_sleep(POLL_SLEEP);
      }
      __builtin_amdgcn_raw_buffer_store_b128(v, myout, (u32)(256 * fp + 32 * frt + row) * 512u + (u32)(64 * ft + col) * 2u, 0, 0);
    }
  }
  STAMP(8);
}

}

extern "C" void kernel_launch(int rank, int nranks, ncclComm_t comm,
                              hipStream_t stream,
                              void* const* d_in, const int* in_sizes, int n_in,
                              void* d_out, int out_size,
                              const int* in_dtypes, int out_dtype,
                              void* const* d_out_all,
                              void* const* d_ws_all, size_t ws_bytes) {
  (void)comm; (void)in_dtypes; (void)out_dtype; (void)d_out;
  if (nranks != NR || n_in != 2 || in_sizes[0] != 256 * 2048 || in_sizes[1] != 2048 * 256 ||
      out_size != 2048 * 256 || ws_bytes < (size_t)FLAG_OFF + FLAG_BYTES) {
    fprintf(stderr, "kernel_launch: unexpected problem shape\n");
    return;
  }
  Params p;
  for (int i = 0; i < NR; ++i) { p.ws[i] = (char*)d_ws_all[i]; p.out[i] = (char*)d_out_all[i]; }
  p.x = (const char*)d_in[0];
  p.w = (const char*)d_in[1];
  p.rank = rank;
  p.myws = p.ws[rank];
  p.myout = p.out[rank];
  ag_gemm_kernel<<<256, 320, 0, stream>>>(p);
  HIP_CHECK(hipGetLastError());
}
```

```cpp
#include <hip/hip_runtime.h>
#include <rccl/rccl.h>
#include <cstdio>

#define HIP_CHECK(expr) do { hipError_t _e = (expr); if (_e != hipSuccess) { \
    fprintf(stderr, "HIP error %s at %s:%d\n", hipGetErrorString(_e), __FILE__, __LINE__); \
    return; } } while (0)

namespace {

typedef unsigned u32;
typedef unsigned long long u64;
typedef u32 u32x4 __attribute__((ext_vector_type(4)));
typedef __bf16 bf16x8 __attribute__((ext_vector_type(8)));
typedef float f32x4 __attribute__((ext_vector_type(4)));
typedef unsigned short u16x2 __attribute__((ext_vector_type(2)));

constexpr int NR = 8;
constexpr u32 KIB = 1024u;
constexpr u32 W_SLOT = 256u * KIB;
constexpr u32 X_RT = 128u * KIB;
constexpr u32 XP_OFF = 12u * W_SLOT;
constexpr u32 XE_OFF = XP_OFF + 5u * X_RT;
constexpr u32 BUF_BYTES = XE_OFF + 6u * X_RT;
constexpr u32 OWNW_OFF = 3u * BUF_BYTES;
constexpr u32 OUTS_OFF = OWNW_OFF + 4u * W_SLOT;
constexpr u32 POISON_CHUNKS = BUF_BYTES / 16u;
constexpr u32 FLAG_OFF = 16u << 20;
constexpr u32 CTR_OFF = 0;
constexpr u32 TMO_OFF = 32u * KIB;
constexpr u32 LNK_OFF = 48u * KIB;
constexpr u32 WF_OFF = 36u * KIB;
constexpr u32 XPF_OFF = 40u * KIB;
constexpr u32 XEF_OFF = 44u * KIB;
constexpr u32 DBG_OFF = 256u * KIB;
constexpr u32 FLAG_BYTES = 1u << 20;
constexpr u32 SHARD_BYTES = 1u << 20;
constexpr u64 DEADLINE = 100000000ull;
constexpr int AUX_SYS = 17;
#define POLL_WIN 4
#define BULK_AUX 17
#define POISON_EARLY 0
#define POISON_AUX 17
#define PACE 2
#define PROBE_MASK 3
#define POLL_SLEEP 1
constexpr int LDS_ROW = 68;

struct Params {
  char* myws;
  char* myout;
  char* ws[NR];
  char* out[NR];
  const char* x;
  const char* w;
  int rank;
};

__device__ __forceinline__ u64 now() { return __builtin_amdgcn_s_memrealtime(); }
__device__ __forceinline__ int par(int v) { return __builtin_popcount(v) & 1; }
__device__ __forceinline__ int fmask(int t) { return t == 3 ? 7 : (1 << t); }
__device__ __forceinline__ int ker_index(int t, int d) {
  return t == 0 ? (d >> 1) : t == 1 ? ((d & 1) | ((d >> 2) << 1)) : t == 2 ? (d & 3) : (d >> 1);
}
__device__ __forceinline__ int ker_elem(int t, int j) {
  return t == 0 ? (j << 1) : t == 1 ? ((j & 1) | ((j >> 1) << 2)) : t == 2 ? j : ((j << 1) | par(j));
}
__device__ __forceinline__ int pair_of(int d) { return (d == 1 || d == 6) ? 0 : (d == 2 || d == 5) ? 1 : 2; }
__device__ __forceinline__ __amdgpu_buffer_rsrc_t rsrc(const void* p, u32 bytes) {
  return __builtin_amdgcn_make_buffer_rsrc(const_cast<void*>(p), 0, (int)bytes, 0x00020000);
}
__device__ __forceinline__ u32 ld_sys(const u32* p) {
  return __hip_atomic_load(p, __ATOMIC_RELAXED, __HIP_MEMORY_SCOPE_SYSTEM);
}
__device__ __forceinline__ void st_sys(u32* p, u32 v) {
  __hip_atomic_store(p, v, __ATOMIC_RELAXED, __HIP_MEMORY_SCOPE_SYSTEM);
}
__device__ __forceinline__ void drain() { asm volatile("s_waitcnt vmcnt(0)" ::: "memory"); }
__device__ __forceinline__ u32 pk_min(u32 a, u32 b) {
  return __builtin_bit_cast(u32, __builtin_elementwise_min(__builtin_bit_cast(u16x2, a), __builtin_bit_cast(u16x2, b)));
}
__device__ __forceinline__ u32 pk_max(u32 a, u32 b) {
  return __builtin_bit_cast(u32, __builtin_elementwise_max(__builtin_bit_cast(u16x2, a), __builtin_bit_cast(u16x2, b)));
}
__device__ __forceinline__ u32 unpoison(u32 v) { return (((pk_min(v, 0x00010001u) ^ 0x00010001u) << 15) | v); }

#define STAMP(i) do { } while (0)

struct Waiter {
  u32* tmo;
  u32 call;
  u64 t0;
  u32 spins;
  bool dead;
  __device__ __forceinline__ bool expired() {
    if (dead) return true;
    ++spins;
    if ((spins & 15u) == 0u) {
      u32 t = __hip_atomic_load(tmo, __ATOMIC_RELAXED, __HIP_MEMORY_SCOPE_AGENT);
      t = __builtin_amdgcn_readfirstlane(t);
      if (t == call) dead = true;
      else if (now() - t0 > DEADLINE) {
        __hip_atomic_store(tmo, call, __ATOMIC_RELAXED, __HIP_MEMORY_SCOPE_AGENT);
        dead = true;
      }
    }
    return dead;
  }
};

__global__ void __launch_bounds__(512) ag_gemm_kernel(Params p) {
  __shared__ __attribute__((aligned(16))) float red[2 * 8 * 32 * LDS_ROW];
  __shared__ u32 deadf[8];
  const int lane = threadIdx.x & 63;
  const int wv = __builtin_amdgcn_readfirstlane(threadIdx.x >> 6);
  const int c = blockIdx.x;
  const int xcd = c & 7, li = c >> 3;
  const int r = p.rank;
  char* myws = p.myws;
  u32* ctr = (u32*)(myws + FLAG_OFF + CTR_OFF + (u32)c * 128u);
  u32* tmo = (u32*)(myws + FLAG_OFF + TMO_OFF);
  STAMP(0);
  const u32 ctrv = __hip_atomic_load(ctr, __ATOMIC_RELAXED, __HIP_MEMORY_SCOPE_AGENT);
  const int g = lane >> 4, l16 = lane & 15;
#define GET_CALL()                                                             \
  const u32 call = __builtin_amdgcn_readfirstlane(ctrv) + 1u;                  \
  const u32 parb = (call % 3u) * BUF_BYTES;

#define POISON()                                                               \
  if (call >= 2u) {                                                            \
    const __amdgpu_buffer_rsrc_t pr = rsrc(myws + ((call + 2u) % 3u) * BUF_BYTES, BUF_BYTES); \
    const u32 per = POISON_CHUNKS / 256u;                                      \
    const u32 base = (u32)(idx * 8 + L) * per;                                 \
    _Pragma("unroll") for (u32 k = 0; k < (per + 63u) / 64u; ++k) {            \
      const u32 ch = k * 64u + (u32)lane;                                      \
      if (ch < per)                                                            \
        __builtin_amdgcn_raw_buffer_store_b128(u32x4{0u, 0u, 0u, 0u}, pr, (base + ch) * 16u, 0, POISON_AUX); \
    }                                                                          \
  }

  if (li < 8 && wv < 4) {
    const int L = (xcd & 1) * 4 + wv;
    const int idx = li * 4 + (xcd >> 1);
    const int sub = idx & 1, kr = idx >> 1;
    const int dst = r ^ L;
    char* dws = p.ws[dst];
    u32* dflag = (u32*)(dws + FLAG_OFF);
    const __amdgpu_buffer_rsrc_t xr = rsrc(p.x, SHARD_BYTES);
    const __amdgpu_buffer_rsrc_t wr = rsrc(p.w, SHARD_BYTES);
    __builtin_amdgcn_s_setprio(3);
    const u32 w_lane = (u32)(8 * g) * 512u + (u32)(2 * l16) * 2u;
    const u32 x_lane = (u32)l16 * 4096u + (u32)(8 * g) * 2u;
#define LOAD_WH(v, t, cb, kr_, hf)                                             \
    _Pragma("unroll") for (int i = 0; i < 16; ++i)                             \
      v[i] = __builtin_amdgcn_raw_buffer_load_b32(wr, w_lane + (u32)(t) * 128u + (u32)(cb) * 64u + \
                 (u32)(128 * (kr_) + 64 * (hf) + (i >> 3) * 32 + (i & 7)) * 512u, 0, 0);
#define STORE_WH(v, off)                                                       \
    _Pragma("unroll") for (int ks = 0; ks < 2; ++ks) {                         \
      u32x4 e, o;                                                              \
      _Pragma("unroll") for (int m = 0; m < 4; ++m) {                          \
        e[m] = unpoison(__builtin_amdgcn_perm(v[ks * 8 + 2 * m + 1], v[ks * 8 + 2 * m], 0x05040100u)); \
        o[m] = unpoison(__builtin_amdgcn_perm(v[ks * 8 + 2 * m + 1], v[ks * 8 + 2 * m], 0x07060302u)); \
      }                                                                        \
      __builtin_amdgcn_raw_buffer_store_b128(e, dr, (u32)(off) + (u32)lane * 16u + (u32)(ks * 2) * 1024u, 0, AUX_SYS); \
      __builtin_amdgcn_raw_buffer_store_b128(o, dr, (u32)(off) + (u32)lane * 16u + (u32)(ks * 2 + 1) * 1024u, 0, AUX_SYS); \
    }
#define LOAD_XH(v, rt, rb, kr_)                                                \
    _Pragma("unroll") for (int ks = 0; ks < 4; ++ks) {                         \
      const u32x4 q_ = __builtin_amdgcn_raw_buffer_load_b128(xr, x_lane + (u32)(rt) * X_RT + (u32)(rb) * 65536u + (u32)(kr_) * 256u + (u32)ks * 64u, 0, 0); \
      v[ks * 4] = q_[0]; v[ks * 4 + 1] = q_[1]; v[ks * 4 + 2] = q_[2]; v[ks * 4 + 3] = q_[3]; \
    }
#define STORE_XH(v, off)                                                       \
    _Pragma("unroll") for (int ks = 0; ks < 4; ++ks)                           \
      __builtin_amdgcn_raw_buffer_store_b128(u32x4{unpoison(v[ks * 4]), unpoison(v[ks * 4 + 1]), unpoison(v[ks * 4 + 2]), unpoison(v[ks * 4 + 3])}, dr, (u32)(off) + (u32)lane * 16u + (u32)ks * 1024u, 0, AUX_SYS);
#define PUSH_SETUP()                                                           \
    __builtin_amdgcn_sched_barrier(0);                                         \
    GET_CALL();                                                                \
    STAMP(10);                                                                 \
    const __amdgpu_buffer_rsrc_t dr = rsrc(dws + (L == 0 ? OWNW_OFF : parb), BUF_BYTES);
#define WAITCNT_(n) asm volatile("s_waitcnt vmcnt(" #n ")" ::: "memory")
#define WAITCNT(n) do { if ((n) == 0) WAITCNT_(0); else if ((n) == 4) WAITCNT_(4); else if ((n) == 8) WAITCNT_(8); else WAITCNT_(12); } while (0)
    if (L == 0) {
      u32 v[8][16];
#pragma unroll
      for (int t = 0; t < 4; ++t) { LOAD_WH(v[2 * t], t, sub, kr, 0); LOAD_WH(v[2 * t + 1], t, sub, kr, 1); }
      PUSH_SETUP();
#pragma unroll
      for (int t = 0; t < 4; ++t) {
        STORE_WH(v[2 * t], (u32)t * W_SLOT + (u32)(kr * 2 + sub) * 8192u);
        STORE_WH(v[2 * t + 1], (u32)t * W_SLOT + (u32)(kr * 2 + sub) * 8192u + 4096u);
      }
      drain();
      if (lane < 8) st_sys(dflag + (WF_OFF >> 2) + (lane >> 1) * 256 + (kr * 2 + sub) * 2 + (lane & 1), call);
      STAMP(1);
      POISON();
    } else {
      int t0 = 0, t1 = 1;
      if (L != 7) {
        while (par(fmask(t0) & L)) ++t0;
        t1 = t0 + 1; while (par(fmask(t1) & L)) ++t1;
      }
      const int s0 = t0 * 4 + ker_index(t0, L), s1 = t1 * 4 + ker_index(t1, L);
      const int xe_rt = 5 + pair_of(L);
      const u32x4 xs = {(u32)(size_t)p.x, (u32)((size_t)p.x >> 32) & 0xffffu, SHARD_BYTES, 0x00020000u};
      const u32x4 wsr = {(u32)(size_t)p.w, (u32)((size_t)p.w >> 32) & 0xffffu, SHARD_BYTES, 0x00020000u};
      u32x4 pa[4], pb[4];
      u32 off_a, fl_a, off_b, fl_b; bool isw_a, isw_b;
#define ALD1(dst, vo, imm) asm volatile("buffer_load_dword %0, %1, %2, 0 offen offset:" #imm : "=v"(dst) : "v"(vo), "s"(wsr) : "memory")
#define ALD4(dst, vo, imm) asm volatile("buffer_load_dwordx4 %0, %1, %2, 0 offen offset:" #imm : "=v"(dst) : "v"(vo), "s"(xs) : "memory")
#define ALD_W8(P, q, vo)                                                       \
          ALD1(P[q][0], vo, 0); ALD1(P[q][1], vo, 512); ALD1(P[q][2], vo, 1024); ALD1(P[q][3], vo, 1536); \
          ALD1(P[q + 1][0], vo, 2048); ALD1(P[q + 1][1], vo, 2560); ALD1(P[q + 1][2], vo, 3072); ALD1(P[q + 1][3], vo, 3584);
#define ALD_X(P, vo) ALD4(P[0], vo, 0); ALD4(P[1], vo, 64); ALD4(P[2], vo, 128); ALD4(P[3], vo, 192);
#define LOAD_PIECE(P, h_, off_, fl_, isw_)                                     \
      {                                                                        \
        const int hk = (h_) / 10, hj = (h_) - 10 * hk;                         \
        if (L == 7 || hj >= 8) {                                               \
          isw_ = false;                                                        \
          const int rt_ = L == 7 ? (hj >> 1) : xe_rt;                          \
          const u32 vo = x_lane + (u32)rt_ * X_RT + (u32)(hj & 1) * 65536u + (u32)hk * 256u; \
          ALD_X(P, vo);                                                        \
          off_ = (L == 7 ? XP_OFF + (u32)rt_ * X_RT : XE_OFF + (u32)(L - 1) * X_RT) + (u32)hk * 8192u + (u32)(hj & 1) * 4096u; \
          fl_ = (L == 7 ? (XPF_OFF >> 2) + (u32)rt_ * 32u : (XEF_OFF >> 2) + (u32)(L - 1) * 32u) + (u32)hk * 2u + (u32)(hj & 1); \
        } else {                                                               \
          const int tt = hj < 4 ? t0 : t1, sf = hj < 4 ? s0 : s1, cb = (hj >> 1) & 1; \
          isw_ = true;                                                         \
          const u32 vo = w_lane + (u32)tt * 128u + (u32)cb * 64u + (u32)(128 * hk + 64 * (hj & 1)) * 512u; \
          const u32 vo2 = vo + 16384u;                                         \
          ALD_W8(P, 0, vo); ALD_W8(P, 2, vo2);                                 \
          off_ = (u32)(sf - tt - 1) * W_SLOT + (u32)(hk * 2 + cb) * 8192u + (u32)(hj & 1) * 4096u; \
          fl_ = (WF_OFF >> 2) + (u32)sf * 64u + (u32)(hk * 2 + cb) * 2u + (u32)(hj & 1); \
        }                                                                      \
      }
      LOAD_PIECE(pa, idx, off_a, fl_a, isw_a);
      LOAD_PIECE(pb, 32 + idx, off_b, fl_b, isw_b);
      PUSH_SETUP();
      u32* lctr = (u32*)(myws + FLAG_OFF + LNK_OFF) + ((call % 3u) * 8u + (u32)L) * 64u;
      bool have_a = true, have_b = true;
      u32 grab = 0;
#define GRAB() if (lane == 0) asm volatile("global_atomic_add %0, %1, %2, off sc0" : "=v"(grab) : "v"(lctr), "v"(1u) : "memory")
      GRAB();
      asm volatile("s_waitcnt vmcnt(1)" ::: "memory");
#define PUSH_STEP(P, off_p, fl_p, isw_p, have_p)                               \
      {                                                                        \
        asm volatile("" : "+v"(P[0]), "+v"(P[1]), "+v"(P[2]), "+v"(P[3]));     \
        if (isw_p) {                                                           \
          _Pragma("unroll") for (int ks = 0; ks < 2; ++ks) {                   \
            u32x4 e, o;                                                        \
            _Pragma("unroll") for (int m = 0; m < 4; ++m) {                    \
              const u32 lo = P[ks * 2 + (m >> 1)][(2 * m) & 3], hi = P[ks * 2 + (m >> 1)][(2 * m + 1) & 3]; \
              e[m] = unpoison(__builtin_amdgcn_perm(hi, lo, 0x05040100u));     \
              o[m] = unpoison(__builtin_amdgcn_perm(hi, lo, 0x07060302u));     \
            }                                                                  \
            __builtin_amdgcn_raw_buffer_store_b128(e, dr, off_p + (u32)lane * 16u + (u32)(ks * 2) * 1024u, 0, AUX_SYS); \
            __builtin_amdgcn_raw_buffer_store_b128(o, dr, off_p + (u32)lane * 16u + (u32)(ks * 2 + 1) * 1024u, 0, AUX_SYS); \
          }                                                                    \
        } else {                                                               \
          _Pragma("unroll") for (int ks = 0; ks < 4; ++ks)                     \
            __builtin_amdgcn_raw_buffer_store_b128(u32x4{unpoison(P[ks][0]), unpoison(P[ks][1]), unpoison(P[ks][2]), unpoison(P[ks][3])}, \
                                                   dr, off_p + (u32)lane * 16u + (u32)ks * 1024u, 0, AUX_SYS); \
        }                                                                      \
        asm volatile("s_waitcnt vmcnt(4)" : "+v"(grab) :: "memory");           \
        const int h = 64 + (int)__builtin_amdgcn_readfirstlane(grab);          \
        have_p = more && h < 160;                                              \
        more = have_p;                                                         \
        if (have_p) {                                                          \
          GRAB();                                                              \
          LOAD_PIECE(P, h, off_p, fl_p, isw_p);                                \
          if (isw_p) asm volatile("s_waitcnt vmcnt(17)" ::: "memory");         \
          else asm volatile("s_waitcnt vmcnt(5)" ::: "memory");                \
        } else drain();                                                        \
        STAMP(1 + (it < 4 ? it : 4));                                          \
        ++it;                                                                  \
      }
      bool more = true;
      int it = 0;
      for (;;) {
        PUSH_STEP(pa, off_a, fl_a, isw_a, have_a);
        if (!have_b) break;
        PUSH_STEP(pb, off_b, fl_b, isw_b, have_b);
        if (!have_a) break;
      }
      STAMP(6);
      POISON();
    }
    return;
  }

  GET_CALL();
  const u32* myflag = (const u32*)(myws + FLAG_OFF);
  const u32 tag = (call & 1u) ? 0x80008000u : 0u;
  const __amdgpu_buffer_rsrc_t myout = rsrc(p.myout, SHARD_BYTES);
  const int t4 = threadIdx.x & 255;
  const int row = t4 >> 3, col = (t4 & 7) * 8;
  Waiter wt; wt.tmo = tmo; wt.call = call; wt.t0 = now(); wt.spins = 0; wt.dead = false;

  if (li >= 8) {
    if (c >= 64 && c < 72 && threadIdx.x == 0)
      __hip_atomic_store((u32*)(myws + FLAG_OFF + LNK_OFF) + (((call + 1u) % 3u) * 8u + (u32)(c - 64)) * 64u, 0u,
                         __ATOMIC_RELAXED, __HIP_MEMORY_SCOPE_AGENT);
    const int t = xcd >> 1, odd = xcd & 1;
    const bool dbl = li >= 24;
    const int j = dbl ? 3 : 1 + odd;
    const int ti = dbl ? li - 24 + 8 * odd : li - 8;
    const int ss = 4 * t + j;
    const int s = r ^ ker_elem(t, j);
    const bool x_own = ti < 8;
    int prow, rt;
    u32 x_base, x_lane, x_rb, x_ks, x_kr;
    __amdgpu_buffer_rsrc_t xr;
    if (x_own) {
      prow = r; rt = ti;
      xr = rsrc(p.x, SHARD_BYTES);
      x_base = (u32)rt * X_RT; x_lane = (u32)l16 * 4096u + (u32)g * 16u; x_rb = 65536u; x_ks = 64u; x_kr = 256u;
    } else {
      xr = rsrc(myws + parb, BUF_BYTES);
      x_lane = (u32)lane * 16u; x_rb = 4096u; x_ks = 1024u; x_kr = 8192u;
      if (ti < 13) {
        prow = r ^ 7; rt = ti - 8;
        x_base = XP_OFF + (u32)rt * X_RT;
      } else {
        int e = 0, n = ti - 13;
        for (int q = 1; q <= 6; ++q) if (par(fmask(t) & q)) { if (n == 0) { e = q; } --n; }
        prow = r ^ e; rt = 5 + pair_of(e);
        x_base = XE_OFF + (u32)(e - 1) * X_RT;
      }
    }
    const u32 w_off = parb + (u32)(ss - t - 1) * W_SLOT;
    const __amdgpu_buffer_rsrc_t wr = rsrc(myws + w_off, W_SLOT);
    const __amdgpu_buffer_rsrc_t wo = rsrc(myws + OWNW_OFF + (u32)t * W_SLOT, W_SLOT);

    const u32* pollp = nullptr;
    const bool poll_flag = lane >= 16;
    if (lane < 16) {
      const int pc = lane & 7, pkr = 8 * (lane >> 3) + wv;
      if (pc < 2) {
        if (!x_own) pollp = (const u32*)(myws + parb + x_base + (u32)pkr * 8192u + (u32)pc * 4096u + 4092u);
      } else if (pc < 6) {
        pollp = (const u32*)(myws + w_off + (u32)pkr * 16384u + (u32)(pc - 2) * 4096u + 4092u);
      }
    } else if (lane < 24 && dbl) {
      pollp = myflag + (WF_OFF >> 2) + (4 * t) * 64 + (8 * ((lane >> 2) & 1) + wv) * 4 + (lane & 3);
    }

    f32x4 acc[2][2][4];
#pragma unroll
    for (int z = 0; z < 2; ++z)
#pragma unroll
      for (int a = 0; a < 2; ++a)
#pragma unroll
        for (int b = 0; b < 4; ++b) acc[z][a][b] = f32x4{0.f, 0.f, 0.f, 0.f};
    u32x4 xb[8], wb[2][16];

#define LOADS(kr_, AUX)                                                       \
  {                                                                           \
    if (x_own) {                                                              \
      _Pragma("unroll") for (int i = 0; i < 8; ++i)                           \
        xb[i] = __builtin_amdgcn_raw_buffer_load_b128(                        \
            xr, x_lane, x_base + (kr_) * x_kr + (u32)(i >> 2) * x_rb + (u32)(i & 3) * x_ks, 0); \
    } else {                                                                  \
      _Pragma("unroll") for (int i = 0; i < 8; ++i)                           \
        xb[i] = __builtin_amdgcn_raw_buffer_load_b128(                        \
            xr, x_lane, x_base + (kr_) * x_kr + (u32)(i >> 2) * x_rb + (u32)(i & 3) * x_ks, AUX); \
    }                                                                         \
    _Pragma("unroll") for (int i = 0; i < 16; ++i)                            \
      wb[0][i] = __builtin_amdgcn_raw_buffer_load_b128(                       \
          wr, (u32)lane * 16u + (u32)(i & 3) * 1024u, (kr_) * 16384u + (u32)(i >> 2) * 4096u, AUX); \
  }
#define COMPUTE(z)                                                            \
  {                                                                           \
    _Pragma("unroll") for (int ks = 0; ks < 4; ++ks)                          \
    _Pragma("unroll") for (int rb = 0; rb < 2; ++rb)                          \
    _Pragma("unroll") for (int cf = 0; cf < 4; ++cf)                          \
      acc[z][rb][cf] = __builtin_amdgcn_mfma_f32_16x16x32_bf16(               \
          __builtin_bit_cast(bf16x8, wb[z][(cf >> 1) * 8 + ks * 2 + (cf & 1)]), \
          __builtin_bit_cast(bf16x8, xb[rb * 4 + ks]), acc[z][rb][cf], 0, 0, 0); \
  }
#define POLL() pv = pollp ? ld_sys(pollp) : (poll_flag ? call : 1u)
    u32 pv;
    POLL();
    STAMP(9);
#pragma unroll 1
    for (int st = 0; st < 2; ++st) {
      const u32 need = (0x3fu << (8 * st)) | (0xfu << (16 + 4 * st));
      for (;;) {
        const bool ok_ = poll_flag ? ((int)(pv - call) >= 0) : (pv != 0u);
        const u32 m_ = (u32)__builtin_amdgcn_ballot_w64(ok_);
        if ((m_ & need) == need) break;
        if (wt.expired()) break;
        __builtin_amdgcn_s_sleep(POLL_SLEEP);
        POLL();
      }
      const u32 kr = 8u * (u32)st + (u32)wv;
      LOADS(kr, BULK_AUX);
      if (dbl) {
#pragma unroll
        for (int i = 0; i < 16; ++i)
          wb[1][i] = __builtin_amdgcn_raw_buffer_load_b128(wo, (u32)lane * 16u + (u32)(i & 3) * 1024u, kr * 16384u + (u32)(i >> 2) * 4096u, BULK_AUX);
      }
      if (st == 0) { POLL(); STAMP(1); } else STAMP(3);
      for (;;) {
        u32 mx_ = ~0u, mw_ = ~0u;
#pragma unroll
        for (int i = 0; i < 8; ++i)
#pragma unroll
          for (int q = 0; q < 4; ++q) mx_ = pk_min(mx_, xb[i][q]);
#pragma unroll
        for (int i = 0; i < 16; ++i)
#pragma unroll
          for (int q = 0; q < 4; ++q) mw_ = pk_min(mw_, wb[0][i][q]);
        if (x_own) mx_ = ~0u;
        const u32 m_ = pk_min(mx_, mw_);
        const bool bad_ = (m_ & 0xffffu) == 0u || (m_ >> 16) == 0u;
        if (__builtin_amdgcn_ballot_w64(bad_) == 0) break;
        if (wt.expired()) break;
        __builtin_amdgcn_s_sleep(POLL_SLEEP);
        LOADS(kr, AUX_SYS);
      }
      COMPUTE(0);
      if (dbl) COMPUTE(1);
      STAMP(2 + 2 * st);
    }
    STAMP(5);

#pragma unroll
    for (int z = 0; z < 2; ++z) {
      if (z == 1 && !dbl) break;
      float* my = red + (z * 8 + wv) * 32 * LDS_ROW;
#pragma unroll
      for (int rb = 0; rb < 2; ++rb)
#pragma unroll
        for (int cb = 0; cb < 2; ++cb) {
          float* q = my + (16 * rb + l16) * LDS_ROW + 32 * cb + 8 * g;
          const f32x4 e = acc[z][rb][cb * 2], o = acc[z][rb][cb * 2 + 1];
          *(f32x4*)q = f32x4{e[0], o[0], e[1], o[1]};
          *(f32x4*)(q + 4) = f32x4{e[2], o[2], e[3], o[3]};
        }
    }
    if (lane == 0) deadf[wv] = wt.dead ? 1u : 0u;
    __syncthreads();
    STAMP(6);
    if (wv < 4 || dbl) {
      const int z = wv >> 2;
      const float* q = red + (z * 8 * 32 + row) * LDS_ROW + col;
      f32x4 a = *(const f32x4*)q, b = *(const f32x4*)(q + 4);
#pragma unroll
      for (int w2 = 1; w2 < 8; ++w2) {
        a += *(const f32x4*)(q + w2 * 32 * LDS_ROW);
        b += *(const f32x4*)(q + w2 * 32 * LDS_ROW + 4);
      }
      float y[8] = {a[0], a[1], a[2], a[3], b[0], b[1], b[2], b[3]};
      u32 h[8];
#pragma unroll
      for (int i = 0; i < 8; ++i) {
        const float v = y[i] < 0.f ? 0.f : y[i];
        const u32 u = __builtin_bit_cast(u32, v);
        h[i] = (v != v) ? 0x7fc0u : (((u + 0x7fffu + ((u >> 16) & 1u)) >> 16) & 0x7fffu);
      }
      u32 anydead = 0;
#pragma unroll
      for (int i = 0; i < 8; ++i) anydead |= deadf[i];
      u32x4 o;
      if (anydead) o = u32x4{0x7fc07fc0u, 0x7fc07fc0u, 0x7fc07fc0u, 0x7fc07fc0u};
      else o = u32x4{h[0] | (h[1] << 16), h[2] | (h[3] << 16), h[4] | (h[5] << 16), h[6] | (h[7] << 16)};
      if (z == 1) {
        __builtin_amdgcn_raw_buffer_store_b128(o, myout, (u32)(256 * prow + 32 * rt + row) * 512u + (u32)(64 * t + col) * 2u, 0, 0);
      } else {
        const __amdgpu_buffer_rsrc_t orr = rsrc(p.ws[s] + OUTS_OFF, 1u << 20);
        __builtin_amdgcn_raw_buffer_store_b128(o | tag, orr, (u32)((8 * prow + rt) * 4 + t) * 4096u + (u32)t4 * 16u, 0, AUX_SYS);
      }
    }
    STAMP(7);
  } else {
    __syncthreads();
  }
  if (threadIdx.x == 256) __hip_atomic_store(ctr, call, __ATOMIC_RELAXED, __HIP_MEMORY_SCOPE_AGENT);
  if (wv >= 4) {
    const int ft = c & 3, frt = (c >> 2) & 7, fp = c >> 5;
    const int fd = fp ^ r;
    const bool local = par(fmask(ft) & fd) == 0 ? (fd == 0)
                       : frt < 5 ? (fd == 7) : (fd != 7 && pair_of(fd) == frt - 5);
    if (!local) {
      const __amdgpu_buffer_rsrc_t ir = rsrc(myws + OUTS_OFF, 1u << 20);
      u32x4 v;
      for (;;) {
        v = __builtin_amdgcn_raw_buffer_load_b128(ir, (u32)c * 4096u + (u32)t4 * 16u, 0, AUX_SYS);
        const u32 bad = ((v[0] ^ tag) | (v[1] ^ tag) | (v[2] ^ tag) | (v[3] ^ tag)) & 0x80008000u;
        if (__builtin_amdgcn_ballot_w64(bad != 0u) == 0) { v &= 0x7fff7fffu; break; }
        if (wt.expired()) { v = u32x4{0x7fc07fc0u, 0x7fc07fc0u, 0x7fc07fc0u, 0x7fc07fc0u}; break; }
        __builtin_amdgcn_s_sleep(POLL_SLEEP);
      }
      __builtin_amdgcn_raw_buffer_store_b128(v, myout, (u32)(256 * fp + 32 * frt + row) * 512u + (u32)(64 * ft + col) * 2u, 0, 0);
    }
  }
  STAMP(8);
}

}

extern "C" void kernel_launch(int rank, int nranks, ncclComm_t comm,
                              hipStream_t stream,
                              void* const* d_in, const int* in_sizes, int n_in,
                              void* d_out, int out_size,
                              const int* in_dtypes, int out_dtype,
                              void* const* d_out_all,
                              void* const* d_ws_all, size_t ws_bytes) {
  (void)comm; (void)in_dtypes; (void)out_dtype; (void)d_out;
  if (nranks != NR || n_in != 2 || in_sizes[0] != 256 * 2048 || in_sizes[1] != 2048 * 256 ||
      out_size != 2048 * 256 || ws_bytes < (size_t)FLAG_OFF + FLAG_BYTES) {
    fprintf(stderr, "kernel_launch: unexpected problem shape\n");
    return;
  }
  Params p;
  for (int i = 0; i < NR; ++i) { p.ws[i] = (char*)d_ws_all[i]; p.out[i] = (char*)d_out_all[i]; }
  p.x = (const char*)d_in[0];
  p.w = (const char*)d_in[1];
  p.rank = rank;
  p.myws = p.ws[rank];
  p.myout = p.out[rank];
  ag_gemm_kernel<<<256, 512, 0, stream>>>(p);
  HIP_CHECK(hipGetLastError());
}
```

```cpp
#include <hip/hip_runtime.h>
#include <rccl/rccl.h>
#include <cstdio>

#define HIP_CHECK(expr) do { hipError_t _e = (expr); if (_e != hipSuccess) { \
    fprintf(stderr, "HIP error %s at %s:%d\n", hipGetErrorString(_e), __FILE__, __LINE__); \
    return; } } while (0)

namespace {

typedef unsigned u32;
typedef unsigned long long u64;
typedef u32 u32x4 __attribute__((ext_vector_type(4)));
typedef __bf16 bf16x8 __attribute__((ext_vector_type(8)));
typedef float f32x4 __attribute__((ext_vector_type(4)));
typedef unsigned short u16x2 __attribute__((ext_vector_type(2)));

constexpr int NR = 8;
constexpr u32 KIB = 1024u;
constexpr u32 W_SLOT = 256u * KIB;
constexpr u32 X_RT = 128u * KIB;
constexpr u32 XP_OFF = 12u * W_SLOT;
constexpr u32 XE_OFF = XP_OFF + 5u * X_RT;
constexpr u32 BUF_BYTES = XE_OFF + 6u * X_RT;
constexpr u32 OWNW_OFF = 3u * BUF_BYTES;
constexpr u32 OUTS_OFF = OWNW_OFF + 4u * W_SLOT;
constexpr u32 POISON_CHUNKS = BUF_BYTES / 16u;
constexpr u32 FLAG_OFF = 16u << 20;
constexpr u32 CTR_OFF = 0;
constexpr u32 TMO_OFF = 32u * KIB;
constexpr u32 LNK_OFF = 48u * KIB;
constexpr u32 WF_OFF = 36u * KIB;
constexpr u32 XPF_OFF = 40u * KIB;
constexpr u32 XEF_OFF = 44u * KIB;
constexpr u32 DBG_OFF = 256u * KIB;
constexpr u32 FLAG_BYTES = 1u << 20;
constexpr u32 SHARD_BYTES = 1u << 20;
constexpr u64 DEADLINE = 100000000ull;
constexpr int AUX_SYS = 17;
#define POLL_WIN 4
#define BULK_AUX 17
#define POISON_EARLY 0
#define POISON_AUX 17
#define PACE 2
#define PROBE_MASK 3
#define POLL_SLEEP 1
constexpr int LDS_ROW = 68;

struct Params {
  char* myws;
  char* myout;
  char* ws[NR];
  char* out[NR];
  const char* x;
  const char* w;
  int rank;
};

__device__ __forceinline__ u64 now() { return __builtin_amdgcn_s_memrealtime(); }
__device__ __forceinline__ int par(int v) { return __builtin_popcount(v) & 1; }
__device__ __forceinline__ int fmask(int t) { return t == 3 ? 7 : (1 << t); }
__device__ __forceinline__ int ker_index(int t, int d) {
  return t == 0 ? (d >> 1) : t == 1 ? ((d & 1) | ((d >> 2) << 1)) : t == 2 ? (d & 3) : (d >> 1);
}
__device__ __forceinline__ int ker_elem(int t, int j) {
  return t == 0 ? (j << 1) : t == 1 ? ((j & 1) | ((j >> 1) << 2)) : t == 2 ? j : ((j << 1) | par(j));
}
__device__ __forceinline__ int pair_of(int d) { return (d == 1 || d == 6) ? 0 : (d == 2 || d == 5) ? 1 : 2; }
__device__ __forceinline__ __amdgpu_buffer_rsrc_t rsrc(const void* p, u32 bytes) {
  return __builtin_amdgcn_make_buffer_rsrc(const_cast<void*>(p), 0, (int)bytes, 0x00020000);
}
__device__ __forceinline__ u32 ld_sys(const u32* p) {
  return __hip_atomic_load(p, __ATOMIC_RELAXED, __HIP_MEMORY_SCOPE_SYSTEM);
}
__device__ __forceinline__ void st_sys(u32* p, u32 v) {
  __hip_atomic_store(p, v, __ATOMIC_RELAXED, __HIP_MEMORY_SCOPE_SYSTEM);
}
__device__ __forceinline__ void drain() { asm volatile("s_waitcnt vmcnt(0)" ::: "memory"); }
__device__ __forceinline__ u32 pk_min(u32 a, u32 b) {
  return __builtin_bit_cast(u32, __builtin_elementwise_min(__builtin_bit_cast(u16x2, a), __builtin_bit_cast(u16x2, b)));
}
__device__ __forceinline__ u32 pk_max(u32 a, u32 b) {
  return __builtin_bit_cast(u32, __builtin_elementwise_max(__builtin_bit_cast(u16x2, a), __builtin_bit_cast(u16x2, b)));
}
__device__ __forceinline__ u32 unpoison(u32 v) { return (((pk_min(v, 0x00010001u) ^ 0x00010001u) << 15) | v); }

#define STAMP(i) do { } while (0)

struct Waiter {
  u32* tmo;
  u32 call;
  u64 t0;
  u32 spins;
  bool dead;
  __device__ __forceinline__ bool expired() {
    if (dead) return true;
    ++spins;
    if ((spins & 15u) == 0u) {
      u32 t = __hip_atomic_load(tmo, __ATOMIC_RELAXED, __HIP_MEMORY_SCOPE_AGENT);
      t = __builtin_amdgcn_readfirstlane(t);
      if (t == call) dead = true;
      else if (now() - t0 > DEADLINE) {
        __hip_atomic_store(tmo, call, __ATOMIC_RELAXED, __HIP_MEMORY_SCOPE_AGENT);
        dead = true;
      }
    }
    return dead;
  }
};

__global__ void __launch_bounds__(320) ag_gemm_kernel(Params p) {
  __shared__ __attribute__((aligned(16))) float red[4 * 32 * LDS_ROW];
  __shared__ u32 deadf[4];
  const int lane = threadIdx.x & 63;
  const int wv = __builtin_amdgcn_readfirstlane(threadIdx.x >> 6);
  const int c = blockIdx.x;
  const int r = p.rank;
  char* myws = p.myws;
  u32* ctr = (u32*)(myws + FLAG_OFF + CTR_OFF + (u32)c * 128u);
  u32* tmo = (u32*)(myws + FLAG_OFF + TMO_OFF);
  STAMP(0);
  const u32 ctrv = __hip_atomic_load(ctr, __ATOMIC_RELAXED, __HIP_MEMORY_SCOPE_AGENT);
  const int g = lane >> 4, l16 = lane & 15;
#define GET_CALL()                                                             \
  const u32 call = __builtin_amdgcn_readfirstlane(ctrv) + 1u;                  \
  const u32 parb = (call % 3u) * BUF_BYTES;

#define POISON()                                                               \
  if (call >= 2u) {                                                            \
    const __amdgpu_buffer_rsrc_t pr = rsrc(myws + ((call + 2u) % 3u) * BUF_BYTES, BUF_BYTES); \
    const u32 per = POISON_EARLY ? POISON_CHUNKS / 1024u : POISON_CHUNKS / 256u; \
    const u32 base = (POISON_EARLY ? (u32)(c * 4 + wv) : (u32)c) * per;        \
    _Pragma("unroll") for (u32 k = 0; k < (per + 63u) / 64u; ++k) {            \
      const u32 ch = k * 64u + (u32)lane;                                      \
      if (ch < per)                                                            \
        __builtin_amdgcn_raw_buffer_store_b128(u32x4{0u, 0u, 0u, 0u}, pr, (base + ch) * 16u, 0, POISON_AUX); \
    }                                                                          \
  }

  if (wv == 4) {
    const int L = (c + (c >> 3)) & 7;
    const int idx = c >> 3;
    const int sub = idx & 1, kr = idx >> 1;
    const int dst = r ^ L;
    char* dws = p.ws[dst];
    u32* dflag = (u32*)(dws + FLAG_OFF);
    const __amdgpu_buffer_rsrc_t xr = rsrc(p.x, SHARD_BYTES);
    const __amdgpu_buffer_rsrc_t wr = rsrc(p.w, SHARD_BYTES);
    __builtin_amdgcn_s_setprio(3);
    const u32 w_lane = (u32)(8 * g) * 512u + (u32)(2 * l16) * 2u;
    const u32 x_lane = (u32)l16 * 4096u + (u32)(8 * g) * 2u;
#define LOAD_WH(v, t, cb, kr_, hf)                                             \
    _Pragma("unroll") for (int i = 0; i < 16; ++i)                             \
      v[i] = __builtin_amdgcn_raw_buffer_load_b32(wr, w_lane + (u32)(t) * 128u + (u32)(cb) * 64u + \
                 (u32)(128 * (kr_) + 64 * (hf) + (i >> 3) * 32 + (i & 7)) * 512u, 0, 0);
#define STORE_WH(v, off)                                                       \
    _Pragma("unroll") for (int ks = 0; ks < 2; ++ks) {                         \
      u32x4 e, o;                                                              \
      _Pragma("unroll") for (int m = 0; m < 4; ++m) {                          \
        e[m] = unpoison(__builtin_amdgcn_perm(v[ks * 8 + 2 * m + 1], v[ks * 8 + 2 * m], 0x05040100u)); \
        o[m] = unpoison(__builtin_amdgcn_perm(v[ks * 8 + 2 * m + 1], v[ks * 8 + 2 * m], 0x07060302u)); \
      }                                                                        \
      __builtin_amdgcn_raw_buffer_store_b128(e, dr, (u32)(off) + (u32)lane * 16u + (u32)(ks * 2) * 1024u, 0, AUX_SYS); \
      __builtin_amdgcn_raw_buffer_store_b128(o, dr, (u32)(off) + (u32)lane * 16u + (u32)(ks * 2 + 1) * 1024u, 0, AUX_SYS); \
    }
#define LOAD_XH(v, rt, rb, kr_)                                                \
    _Pragma("unroll") for (int ks = 0; ks < 4; ++ks) {                         \
      const u32x4 q_ = __builtin_amdgcn_raw_buffer_load_b128(xr, x_lane + (u32)(rt) * X_RT + (u32)(rb) * 65536u + (u32)(kr_) * 256u + (u32)ks * 64u, 0, 0); \
      v[ks * 4] = q_[0]; v[ks * 4 + 1] = q_[1]; v[ks * 4 + 2] = q_[2]; v[ks * 4 + 3] = q_[3]; \
    }
#define STORE_XH(v, off)                                                       \
    _Pragma("unroll") for (int ks = 0; ks < 4; ++ks)                           \
      __builtin_amdgcn_raw_buffer_store_b128(u32x4{unpoison(v[ks * 4]), unpoison(v[ks * 4 + 1]), unpoison(v[ks * 4 + 2]), unpoison(v[ks * 4 + 3])}, dr, (u32)(off) + (u32)lane * 16u + (u32)ks * 1024u, 0, AUX_SYS);
#define PUSH_SETUP()                                                           \
    __builtin_amdgcn_sched_barrier(0);                                         \
    GET_CALL();                                                                \
    STAMP(10);                                                                 \
    const __amdgpu_buffer_rsrc_t dr = rsrc(dws + (L == 0 ? OWNW_OFF : parb), BUF_BYTES);
#define WAITCNT_(n) asm volatile("s_waitcnt vmcnt(" #n ")" ::: "memory")
#define WAITCNT(n) do { if ((n) == 0) WAITCNT_(0); else if ((n) == 4) WAITCNT_(4); else if ((n) == 8) WAITCNT_(8); else WAITCNT_(12); } while (0)
    if (L == 0) {
      u32 v[8][16];
#pragma unroll
      for (int t = 0; t < 4; ++t) { LOAD_WH(v[2 * t], t, sub, kr, 0); LOAD_WH(v[2 * t + 1], t, sub, kr, 1); }
      PUSH_SETUP();
#pragma unroll
      for (int t = 0; t < 4; ++t) {
        STORE_WH(v[2 * t], (u32)t * W_SLOT + (u32)(kr * 2 + sub) * 8192u);
        STORE_WH(v[2 * t + 1], (u32)t * W_SLOT + (u32)(kr * 2 + sub) * 8192u + 4096u);
      }
      drain();
      if (lane < 8) st_sys(dflag + (WF_OFF >> 2) + (lane >> 1) * 256 + (kr * 2 + sub) * 2 + (lane & 1), call);
      STAMP(1);
      POISON();
    } else {
      int t0 = 0, t1 = 1;
      if (L != 7) {
        while (par(fmask(t0) & L)) ++t0;
        t1 = t0 + 1; while (par(fmask(t1) & L)) ++t1;
      }
      const int s0 = t0 * 4 + ker_index(t0, L), s1 = t1 * 4 + ker_index(t1, L);
      const int xe_rt = 5 + pair_of(L);
      const u32x4 xs = {(u32)(size_t)p.x, (u32)((size_t)p.x >> 32) & 0xffffu, SHARD_BYTES, 0x00020000u};
      const u32x4 wsr = {(u32)(size_t)p.w, (u32)((size_t)p.w >> 32) & 0xffffu, SHARD_BYTES, 0x00020000u};
      u32x4 pa[4], pb[4];
      u32 off_a, fl_a, off_b, fl_b; bool isw_a, isw_b;
#define ALD1(dst, vo, imm) asm volatile("buffer_load_dword %0, %1, %2, 0 offen offset:" #imm : "=v"(dst) : "v"(vo), "s"(wsr) : "memory")
#define ALD4(dst, vo, imm) asm volatile("buffer_load_dwordx4 %0, %1, %2, 0 offen offset:" #imm : "=v"(dst) : "v"(vo), "s"(xs) : "memory")
#define ALD_W8(P, q, vo)                                                       \
          ALD1(P[q][0], vo, 0); ALD1(P[q][1], vo, 512); ALD1(P[q][2], vo, 1024); ALD1(P[q][3], vo, 1536); \
          ALD1(P[q + 1][0], vo, 2048); ALD1(P[q + 1][1], vo, 2560); ALD1(P[q + 1][2], vo, 3072); ALD1(P[q + 1][3], vo, 3584);
#define ALD_X(P, vo) ALD4(P[0], vo, 0); ALD4(P[1], vo, 64); ALD4(P[2], vo, 128); ALD4(P[3], vo, 192);
#define LOAD_PIECE(P, h_, off_, fl_, isw_)                                     \
      {                                                                        \
        const int hk = (h_) / 10, hj = (h_) - 10 * hk;                         \
        if (L == 7 || hj >= 8) {                                               \
          isw_ = false;                                                        \
          const int rt_ = L == 7 ? (hj >> 1) : xe_rt;                          \
          const u32 vo = x_lane + (u32)rt_ * X_RT + (u32)(hj & 1) * 65536u + (u32)hk * 256u; \
          ALD_X(P, vo);                                                        \
          off_ = (L == 7 ? XP_OFF + (u32)rt_ * X_RT : XE_OFF + (u32)(L - 1) * X_RT) + (u32)hk * 8192u + (u32)(hj & 1) * 4096u; \
          fl_ = (L == 7 ? (XPF_OFF >> 2) + (u32)rt_ * 32u : (XEF_OFF >> 2) + (u32)(L - 1) * 32u) + (u32)hk * 2u + (u32)(hj & 1); \
        } else {                                                               \
          const int tt = hj < 4 ? t0 : t1, sf = hj < 4 ? s0 : s1, cb = (hj >> 1) & 1; \
          isw_ = true;                                                         \
          const u32 vo = w_lane + (u32)tt * 128u + (u32)cb * 64u + (u32)(128 * hk + 64 * (hj & 1)) * 512u; \
          const u32 vo2 = vo + 16384u;                                         \
          ALD_W8(P, 0, vo); ALD_W8(P, 2, vo2);                                 \
          off_ = (u32)(sf - tt - 1) * W_SLOT + (u32)(hk * 2 + cb) * 8192u + (u32)(hj & 1) * 4096u; \
          fl_ = (WF_OFF >> 2) + (u32)sf * 64u + (u32)(hk * 2 + cb) * 2u + (u32)(hj & 1); \
        }                                                                      \
      }
      LOAD_PIECE(pa, idx, off_a, fl_a, isw_a);
      LOAD_PIECE(pb, 32 + idx, off_b, fl_b, isw_b);
      PUSH_SETUP();
      u32* lctr = (u32*)(myws + FLAG_OFF + LNK_OFF) + ((call % 3u) * 8u + (u32)L) * 64u;
      bool have_a = true, have_b = true;
      u32 grab = 0;
#define GRAB() if (lane == 0) asm volatile("global_atomic_add %0, %1, %2, off sc0" : "=v"(grab) : "v"(lctr), "v"(1u) : "memory")
      GRAB();
      asm volatile("s_waitcnt vmcnt(1)" ::: "memory");
#define PUSH_STEP(P, off_p, fl_p, isw_p, have_p)                               \
      {                                                                        \
        asm volatile("" : "+v"(P[0]), "+v"(P[1]), "+v"(P[2]), "+v"(P[3]));     \
        if (isw_p) {                                                           \
          _Pragma("unroll") for (int ks = 0; ks < 2; ++ks) {                   \
            u32x4 e, o;                                                        \
            _Pragma("unroll") for (int m = 0; m < 4; ++m) {                    \
              const u32 lo = P[ks * 2 + (m >> 1)][(2 * m) & 3], hi = P[ks * 2 + (m >> 1)][(2 * m + 1) & 3]; \
              e[m] = unpoison(__builtin_amdgcn_perm(hi, lo, 0x05040100u));     \
              o[m] = unpoison(__builtin_amdgcn_perm(hi, lo, 0x07060302u));     \
            }                                                                  \
            __builtin_amdgcn_raw_buffer_store_b128(e, dr, off_p + (u32)lane * 16u + (u32)(ks * 2) * 1024u, 0, AUX_SYS); \
            __builtin_amdgcn_raw_buffer_store_b128(o, dr, off_p + (u32)lane * 16u + (u32)(ks * 2 + 1) * 1024u, 0, AUX_SYS); \
          }                                                                    \
        } else {                                                               \
          _Pragma("unroll") for (int ks = 0; ks < 4; ++ks)                     \
            __builtin_amdgcn_raw_buffer_store_b128(u32x4{unpoison(P[ks][0]), unpoison(P[ks][1]), unpoison(P[ks][2]), unpoison(P[ks][3])}, \
                                                   dr, off_p + (u32)lane * 16u + (u32)ks * 1024u, 0, AUX_SYS); \
        }                                                                      \
        asm volatile("s_waitcnt vmcnt(4)" : "+v"(grab) :: "memory");           \
        const int h = 64 + (int)__builtin_amdgcn_readfirstlane(grab);          \
        have_p = more && h < 160;                                              \
        more = have_p;                                                         \
        if (have_p) {                                                          \
          GRAB();                                                              \
          LOAD_PIECE(P, h, off_p, fl_p, isw_p);                                \
          if (isw_p) asm volatile("s_waitcnt vmcnt(17)" ::: "memory");         \
          else asm volatile("s_waitcnt vmcnt(5)" ::: "memory");                \
        } else drain();                                                        \
        STAMP(1 + (it < 4 ? it : 4));                                          \
        ++it;                                                                  \
      }
      bool more = true;
      int it = 0;
      for (;;) {
        PUSH_STEP(pa, off_a, fl_a, isw_a, have_a);
        if (!have_b) break;
        PUSH_STEP(pb, off_b, fl_b, isw_b, have_b);
        if (!have_a) break;
      }
      STAMP(6);
      POISON();
    }
    return;
  }

  GET_CALL();
  const u32* myflag = (const u32*)(myws + FLAG_OFF);
  if (c < 8 && threadIdx.x == 0)
    __hip_atomic_store((u32*)(myws + FLAG_OFF + LNK_OFF) + (((call + 1u) % 3u) * 8u + (u32)c) * 64u, 0u,
                       __ATOMIC_RELAXED, __HIP_MEMORY_SCOPE_AGENT);
  const int xcd = c & 7, li = c >> 3;
  const int ss = 2 * xcd + (li >> 4);
  const int ti = li & 15;
  const int t = ss >> 2;
  const int s = r ^ ker_elem(t, ss & 3);
  const bool w_own = (ss & 3) == 0;
  const bool x_own = ti < 8;
  int prow, rt;
  u32 x_base, x_lane, x_rb, x_ks, x_kr;
  const u32* xflag = nullptr;
  __amdgpu_buffer_rsrc_t xr;
  if (x_own) {
    prow = r; rt = ti;
    xr = rsrc(p.x, SHARD_BYTES);
    x_base = (u32)rt * X_RT; x_lane = (u32)l16 * 4096u + (u32)g * 16u; x_rb = 65536u; x_ks = 64u; x_kr = 256u;
  } else {
    xr = rsrc(myws + parb, BUF_BYTES);
    x_lane = (u32)lane * 16u; x_rb = 4096u; x_ks = 1024u; x_kr = 8192u;
    if (ti < 13) {
      prow = r ^ 7; rt = ti - 8;
      x_base = XP_OFF + (u32)rt * X_RT;
      xflag = myflag + (XPF_OFF >> 2) + rt * 32;
    } else {
      int e = 0, n = ti - 13;
      for (int q = 1; q <= 6; ++q) if (par(fmask(t) & q)) { if (n == 0) { e = q; } --n; }
      prow = r ^ e; rt = 5 + pair_of(e);
      x_base = XE_OFF + (u32)(e - 1) * X_RT;
      xflag = myflag + (XEF_OFF >> 2) + (e - 1) * 32;
    }
  }
  const u32 w_off = w_own ? OWNW_OFF + (u32)t * W_SLOT : parb + (u32)(ss - t - 1) * W_SLOT;
  const __amdgpu_buffer_rsrc_t wr = rsrc(myws + w_off, W_SLOT);
  const u32* wflag = myflag + (WF_OFF >> 2) + ss * 64;

  const u32* pollp = nullptr;
  bool poll_flag = false;
  {
    const int pst = lane >> 3, pc = lane & 7, pkr = 4 * pst + wv;
    if (lane < 32 && pc < 2) {
      if (!x_own) {
        pollp = (const u32*)(myws + parb + x_base + (u32)pkr * 8192u + (u32)pc * 4096u + 4092u);
      }
    } else if (lane < 32 && pc < 6) {
      poll_flag = w_own;
      pollp = poll_flag ? wflag + pkr * 4 + (pc - 2)
                        : (const u32*)(myws + w_off + (u32)pkr * 16384u + (u32)(pc - 2) * 4096u + 4092u);
    }
  }
  const bool use_probe = true;

  Waiter wt; wt.tmo = tmo; wt.call = call; wt.t0 = now(); wt.spins = 0; wt.dead = false;
  u32 done = 0;

  f32x4 acc[2][4];
#pragma unroll
  for (int a = 0; a < 2; ++a)
#pragma unroll
    for (int b = 0; b < 4; ++b) acc[a][b] = f32x4{0.f, 0.f, 0.f, 0.f};
  u32x4 xb[2][8], wb[2][16];

#define LOADS(buf, kr_, AUX)                                                  \
  {                                                                           \
    if (x_own) {                                                              \
      _Pragma("unroll") for (int i = 0; i < 8; ++i)                           \
        xb[buf][i] = __builtin_amdgcn_raw_buffer_load_b128(                   \
            xr, x_base + x_lane + (kr_) * x_kr + (u32)(i >> 2) * x_rb + (u32)(i & 3) * x_ks, 0, 0); \
    } else {                                                                  \
      _Pragma("unroll") for (int i = 0; i < 8; ++i)                           \
        xb[buf][i] = __builtin_amdgcn_raw_buffer_load_b128(                   \
            xr, x_base + x_lane + (kr_) * x_kr + (u32)(i >> 2) * x_rb + (u32)(i & 3) * x_ks, 0, AUX); \
    }                                                                         \
    _Pragma("unroll") for (int i = 0; i < 16; ++i)                            \
      wb[buf][i] = __builtin_amdgcn_raw_buffer_load_b128(                     \
          wr, (u32)lane * 16u + (kr_) * 16384u + (u32)i * 1024u, 0, AUX);     \
  }
#define DBG_POLL(first)
#define POLL() { const int lo_ = __builtin_ctz(~done); \
    pv = (lane >> 3) >= lo_ + POLL_WIN ? (poll_flag ? call - 1u : 0u) : pollp ? ld_sys(pollp) : (poll_flag ? call : 1u); }
  u32 pv = 0, krb[2];
#define VERIFY(buf)                                                           \
  if (use_probe) {                                                            \
    for (;;) {                                                                \
      u32 mx_ = ~0u, mw_ = ~0u;                                               \
      _Pragma("unroll") for (int i = 0; i < 8; ++i)                           \
      _Pragma("unroll") for (int q = 0; q < 4; ++q) mx_ = pk_min(mx_, xb[buf][i][q]); \
      _Pragma("unroll") for (int i = 0; i < 16; ++i)                          \
      _Pragma("unroll") for (int q = 0; q < 4; ++q) mw_ = pk_min(mw_, wb[buf][i][q]); \
      if (x_own) mx_ = ~0u;                                                   \
      const u32 m_ = pk_min(mx_, mw_);                                        \
      const bool bad_ = (m_ & 0xffffu) == 0u || (m_ >> 16) == 0u;             \
      if (__builtin_amdgcn_ballot_w64(bad_) == 0) break;                      \
      if (wt.expired()) break;                                                \
      __builtin_amdgcn_s_sleep(POLL_SLEEP);                                   \
      LOADS(buf, krb[buf], AUX_SYS);                                          \
    }                                                                         \
  }
#define COMPUTE(buf)                                                          \
  {                                                                           \
    _Pragma("unroll") for (int ks = 0; ks < 4; ++ks)                          \
    _Pragma("unroll") for (int rb = 0; rb < 2; ++rb)                          \
    _Pragma("unroll") for (int cf = 0; cf < 4; ++cf)                          \
      acc[rb][cf] = __builtin_amdgcn_mfma_f32_16x16x32_bf16(                  \
          __builtin_bit_cast(bf16x8, wb[buf][(cf >> 1) * 8 + ks * 2 + (cf & 1)]), \
          __builtin_bit_cast(bf16x8, xb[buf][rb * 4 + ks]), acc[rb][cf], 0, 0, 0); \
  }

  POLL();
  STAMP(9);
  for (int rnd = 0; done != 0xfu; ++rnd) {
    u32 avail;
    for (;;) {
      const bool ok_ = poll_flag ? ((int)(pv - call) >= 0) : (pv != 0u);
      const u32 m_ = (u32)__builtin_amdgcn_ballot_w64(ok_);
      avail = 0;
#pragma unroll
      for (int q = 0; q < 4; ++q)
        if (((m_ >> (8 * q)) & 0x3fu) == 0x3fu) avail |= 1u << q;
      avail &= ~done;
      if (avail) break;
      if (wt.expired()) { avail = ~done & 0xfu; break; }
      __builtin_amdgcn_s_sleep(POLL_SLEEP);
      POLL();
    }
    const u32 st0 = (u32)__builtin_ctz(avail);
    avail &= avail - 1u;
    done |= 1u << st0;
    krb[0] = 4u * st0 + (u32)wv;
    LOADS(0, krb[0], BULK_AUX);
    const bool two = avail != 0u;
    if (two) {
      const u32 st1 = (u32)__builtin_ctz(avail);
      done |= 1u << st1;
      krb[1] = 4u * st1 + (u32)wv;
      LOADS(1, krb[1], BULK_AUX);
    }
    if (done != 0xfu) POLL();
    if (rnd == 0) STAMP(1);
    VERIFY(0);
    COMPUTE(0);
    if (two) {
      VERIFY(1);
      COMPUTE(1);
    }
    STAMP(2 + (rnd < 2 ? rnd : 2));
  }
  STAMP(5);

  {
    float* my = red + wv * 32 * LDS_ROW;
#pragma unroll
    for (int rb = 0; rb < 2; ++rb)
#pragma unroll
      for (int cb = 0; cb < 2; ++cb) {
        float* q = my + (16 * rb + l16) * LDS_ROW + 32 * cb + 8 * g;
        const f32x4 e = acc[rb][cb * 2], o = acc[rb][cb * 2 + 1];
        *(f32x4*)q = f32x4{e[0], o[0], e[1], o[1]};
        *(f32x4*)(q + 4) = f32x4{e[2], o[2], e[3], o[3]};
      }
  }
  if (lane == 0) deadf[wv] = wt.dead ? 1u : 0u;
  __syncthreads();
  STAMP(6);
  if (threadIdx.x == 0) __hip_atomic_store(ctr, call, __ATOMIC_RELAXED, __HIP_MEMORY_SCOPE_AGENT);
  const u32 tag = (call & 1u) ? 0x80008000u : 0u;
  const __amdgpu_buffer_rsrc_t myout = rsrc(p.myout, SHARD_BYTES);
  const int row = 8 * wv + (lane >> 3), col = (lane & 7) * 8;
  {
    const float* q = red + row * LDS_ROW + col;
    f32x4 a = *(const f32x4*)q, b = *(const f32x4*)(q + 4);
#pragma unroll
    for (int w2 = 1; w2 < 4; ++w2) {
      a += *(const f32x4*)(q + w2 * 32 * LDS_ROW);
      b += *(const f32x4*)(q + w2 * 32 * LDS_ROW + 4);
    }
    float y[8] = {a[0], a[1], a[2], a[3], b[0], b[1], b[2], b[3]};
    u32 h[8];
#pragma unroll
    for (int i = 0; i < 8; ++i) {
      const float v = y[i] < 0.f ? 0.f : y[i];
      const u32 u = __builtin_bit_cast(u32, v);
      h[i] = (v != v) ? 0x7fc0u : (((u + 0x7fffu + ((u >> 16) & 1u)) >> 16) & 0x7fffu);
    }
    const u32 anydead = deadf[0] | deadf[1] | deadf[2] | deadf[3];
    u32x4 o;
    if (anydead) o = u32x4{0x7fc07fc0u, 0x7fc07fc0u, 0x7fc07fc0u, 0x7fc07fc0u};
    else o = u32x4{h[0] | (h[1] << 16), h[2] | (h[3] << 16), h[4] | (h[5] << 16), h[6] | (h[7] << 16)};
    if (s == r) {
      __builtin_amdgcn_raw_buffer_store_b128(o, myout, (u32)(256 * prow + 32 * rt + row) * 512u + (u32)(64 * t + col) * 2u, 0, 0);
    } else {
      const __amdgpu_buffer_rsrc_t orr = rsrc(p.ws[s] + OUTS_OFF, 1u << 20);
      __builtin_amdgcn_raw_buffer_store_b128(o | tag, orr, (u32)((8 * prow + rt) * 4 + t) * 4096u + (u32)threadIdx.x * 16u, 0, AUX_SYS);
    }
  }
  STAMP(7);
  {
    const int ft = c & 3, frt = (c >> 2) & 7, fp = c >> 5;
    const int fd = fp ^ r;
    const bool local = par(fmask(ft) & fd) == 0 ? (fd == 0)
                       : frt < 5 ? (fd == 7) : (fd != 7 && pair_of(fd) == frt - 5);
    if (!local) {
      const __amdgpu_buffer_rsrc_t ir = rsrc(myws + OUTS_OFF, 1u << 20);
      u32x4 v;
      for (;;) {
        v = __builtin_amdgcn_raw_buffer_load_b128(ir, (u32)c * 4096u + (u32)threadIdx.x * 16u, 0, AUX_SYS);
        const u32 bad = ((v[0] ^ tag) | (v[1] ^ tag) | (v[2] ^ tag) | (v[3] ^ tag)) & 0x80008000u;
        if (__builtin_amdgcn_ballot_w64(bad != 0u) == 0) { v &= 0x7fff7fffu; break; }
        if (wt.expired()) { v = u32x4{0x7fc07fc0u, 0x7fc07fc0u, 0x7fc07fc0u, 0x7fc07fc0u}; break; }
        __builtin_amdgcn_s_sleep(POLL_SLEEP);
      }
      __builtin_amdgcn_raw_buffer_store_b128(v, myout, (u32)(256 * fp + 32 * frt + row) * 512u + (u32)(64 * ft + col) * 2u, 0, 0);
    }
  }
  STAMP(8);
}

}

extern "C" void kernel_launch(int rank, int nranks, ncclComm_t comm,
                              hipStream_t stream,
                              void* const* d_in, const int* in_sizes, int n_in,
                              void* d_out, int out_size,
                              const int* in_dtypes, int out_dtype,
                              void* const* d_out_all,
                              void* const* d_ws_all, size_t ws_bytes) {
  (void)comm; (void)in_dtypes; (void)out_dtype; (void)d_out;
  if (nranks != NR || n_in != 2 || in_sizes[0] != 256 * 2048 || in_sizes[1] != 2048 * 256 ||
      out_size != 2048 * 256 || ws_bytes < (size_t)FLAG_OFF + FLAG_BYTES) {
    fprintf(stderr, "kernel_launch: unexpected problem shape\n");
    return;
  }
  Params p;
  for (int i = 0; i < NR; ++i) { p.ws[i] = (char*)d_ws_all[i]; p.out[i] = (char*)d_out_all[i]; }
  p.x = (const char*)d_in[0];
  p.w = (const char*)d_in[1];
  p.rank = rank;
  p.myws = p.ws[rank];
  p.myout = p.out[rank];
  ag_gemm_kernel<<<256, 320, 0, stream>>>(p);
  HIP_CHECK(hipGetLastError());
}
```

```cpp
#include <hip/hip_runtime.h>
#include <rccl/rccl.h>
#include <cstdio>

#define HIP_CHECK(expr) do { hipError_t _e = (expr); if (_e != hipSuccess) { \
    fprintf(stderr, "HIP error %s at %s:%d\n", hipGetErrorString(_e), __FILE__, __LINE__); \
    return; } } while (0)

namespace {

typedef unsigned u32;
typedef unsigned long long u64;
typedef u32 u32x4 __attribute__((ext_vector_type(4)));
typedef __bf16 bf16x8 __attribute__((ext_vector_type(8)));
typedef float f32x4 __attribute__((ext_vector_type(4)));
typedef unsigned short u16x2 __attribute__((ext_vector_type(2)));

constexpr int NR = 8;
constexpr u32 KIB = 1024u;
constexpr u32 W_SLOT = 256u * KIB;
constexpr u32 X_RT = 128u * KIB;
constexpr u32 XP_OFF = 12u * W_SLOT;
constexpr u32 XE_OFF = XP_OFF + 5u * X_RT;
constexpr u32 BUF_BYTES = XE_OFF + 6u * X_RT;
constexpr u32 OWNW_OFF = 3u * BUF_BYTES;
constexpr u32 OUTS_OFF = OWNW_OFF + 4u * W_SLOT;
constexpr u32 POISON_CHUNKS = BUF_BYTES / 16u;
constexpr u32 FLAG_OFF = 16u << 20;
constexpr u32 CTR_OFF = 0;
constexpr u32 TMO_OFF = 32u * KIB;
constexpr u32 LNK_OFF = 48u * KIB;
constexpr u32 WF_OFF = 36u * KIB;
constexpr u32 XPF_OFF = 40u * KIB;
constexpr u32 XEF_OFF = 44u * KIB;
constexpr u32 DBG_OFF = 256u * KIB;
constexpr u32 FLAG_BYTES = 1u << 20;
constexpr u32 SHARD_BYTES = 1u << 20;
constexpr u64 DEADLINE = 100000000ull;
constexpr int AUX_SYS = 17;
#define POLL_WIN 4
#define BULK_AUX 17
#define POISON_EARLY 0
#define POISON_AUX 17
#define PACE 2
#define PROBE_MASK 3
#define POLL_SLEEP 1
constexpr int LDS_ROW = 68;

struct Params {
  char* myws;
  char* myout;
  char* ws[NR];
  char* out[NR];
  const char* x;
  const char* w;
  int rank;
};

__device__ __forceinline__ u64 now() { return __builtin_amdgcn_s_memrealtime(); }
__device__ __forceinline__ int par(int v) { return __builtin_popcount(v) & 1; }
__device__ __forceinline__ int fmask(int t) { return t == 3 ? 7 : (1 << t); }
__device__ __forceinline__ int ker_index(int t, int d) {
  return t == 0 ? (d >> 1) : t == 1 ? ((d & 1) | ((d >> 2) << 1)) : t == 2 ? (d & 3) : (d >> 1);
}
__device__ __forceinline__ int ker_elem(int t, int j) {
  return t == 0 ? (j << 1) : t == 1 ? ((j & 1) | ((j >> 1) << 2)) : t == 2 ? j : ((j << 1) | par(j));
}
__device__ __forceinline__ int pair_of(int d) { return (d == 1 || d == 6) ? 0 : (d == 2 || d == 5) ? 1 : 2; }
__device__ __forceinline__ __amdgpu_buffer_rsrc_t rsrc(const void* p, u32 bytes) {
  return __builtin_amdgcn_make_buffer_rsrc(const_cast<void*>(p), 0, (int)bytes, 0x00020000);
}
__device__ __forceinline__ u32 ld_sys(const u32* p) {
  return __hip_atomic_load(p, __ATOMIC_RELAXED, __HIP_MEMORY_SCOPE_SYSTEM);
}
__device__ __forceinline__ void st_sys(u32* p, u32 v) {
  __hip_atomic_store(p, v, __ATOMIC_RELAXED, __HIP_MEMORY_SCOPE_SYSTEM);
}
__device__ __forceinline__ void drain() { asm volatile("s_waitcnt vmcnt(0)" ::: "memory"); }
__device__ __forceinline__ u32 pk_min(u32 a, u32 b) {
  return __builtin_bit_cast(u32, __builtin_elementwise_min(__builtin_bit_cast(u16x2, a), __builtin_bit_cast(u16x2, b)));
}
__device__ __forceinline__ u32 pk_max(u32 a, u32 b) {
  return __builtin_bit_cast(u32, __builtin_elementwise_max(__builtin_bit_cast(u16x2, a), __builtin_bit_cast(u16x2, b)));
}
__device__ __forceinline__ u32 unpoison(u32 v) { return (((pk_min(v, 0x00010001u) ^ 0x00010001u) << 15) | v); }

#define STAMP(i) do { } while (0)

struct Waiter {
  u32* tmo;
  u32 call;
  u64 t0;
  u32 spins;
  bool dead;
  __device__ __forceinline__ bool expired() {
    if (dead) return true;
    ++spins;
    if ((spins & 15u) == 0u) {
      u32 t = __hip_atomic_load(tmo, __ATOMIC_RELAXED, __HIP_MEMORY_SCOPE_AGENT);
      t = __builtin_amdgcn_readfirstlane(t);
      if (t == call) dead = true;
      else if (now() - t0 > DEADLINE) {
        __hip_atomic_store(tmo, call, __ATOMIC_RELAXED, __HIP_MEMORY_SCOPE_AGENT);
        dead = true;
      }
    }
    return dead;
  }
};

__global__ void __launch_bounds__(320) ag_gemm_kernel(Params p) {
  __shared__ __attribute__((aligned(16))) float red[4 * 32 * LDS_ROW];
  __shared__ u32 deadf[4];
  const int lane = threadIdx.x & 63;
  const int wv = __builtin_amdgcn_readfirstlane(threadIdx.x >> 6);
  const int c = blockIdx.x;
  const int r = p.rank;
  char* myws = p.myws;
  u32* ctr = (u32*)(myws + FLAG_OFF + CTR_OFF + (u32)c * 128u);
  u32* tmo = (u32*)(myws + FLAG_OFF + TMO_OFF);
  STAMP(0);
  const u32 ctrv = __hip_atomic_load(ctr, __ATOMIC_RELAXED, __HIP_MEMORY_SCOPE_AGENT);
  const int g = lane >> 4, l16 = lane & 15;
#define GET_CALL()                                                             \
  const u32 call = __builtin_amdgcn_readfirstlane(ctrv) + 1u;                  \
  const u32 parb = (call % 3u) * BUF_BYTES;

#define POISON()                                                               \
  if (call >= 2u) {                                                            \
    const __amdgpu_buffer_rsrc_t pr = rsrc(myws + ((call + 2u) % 3u) * BUF_BYTES, BUF_BYTES); \
    const u32 per = POISON_EARLY ? POISON_CHUNKS / 1024u : POISON_CHUNKS / 256u; \
    const u32 base = (POISON_EARLY ? (u32)(c * 4 + wv) : (u32)c) * per;        \
    _Pragma("unroll") for (u32 k = 0; k < (per + 63u) / 64u; ++k) {            \
      const u32 ch = k * 64u + (u32)lane;                                      \
      if (ch < per)                                                            \
        __builtin_amdgcn_raw_buffer_store_b128(u32x4{0u, 0u, 0u, 0u}, pr, (base + ch) * 16u, 0, POISON_AUX); \
    }                                                                          \
  }

  if (wv == 4) {
    const int L = (c + (c >> 3)) & 7;
    const int idx = c >> 3;
    const int sub = idx & 1, kr = idx >> 1;
    const int dst = r ^ L;
    char* dws = p.ws[dst];
    u32* dflag = (u32*)(dws + FLAG_OFF);
    const __amdgpu_buffer_rsrc_t xr = rsrc(p.x, SHARD_BYTES);
    const __amdgpu_buffer_rsrc_t wr = rsrc(p.w, SHARD_BYTES);
    __builtin_amdgcn_s_setprio(3);
    const u32 w_lane = (u32)(8 * g) * 512u + (u32)(2 * l16) * 2u;
    const u32 x_lane = (u32)l16 * 4096u + (u32)(8 * g) * 2u;
#define LOAD_WH(v, t, cb, kr_, hf)                                             \
    _Pragma("unroll") for (int i = 0; i < 16; ++i)                             \
      v[i] = __builtin_amdgcn_raw_buffer_load_b32(wr, w_lane + (u32)(t) * 128u + (u32)(cb) * 64u + \
                 (u32)(128 * (kr_) + 64 * (hf) + (i >> 3) * 32 + (i & 7)) * 512u, 0, 0);
#define STORE_WH(v, off)                                                       \
    _Pragma("unroll") for (int ks = 0; ks < 2; ++ks) {                         \
      u32x4 e, o;                                                              \
      _Pragma("unroll") for (int m = 0; m < 4; ++m) {                          \
        e[m] = unpoison(__builtin_amdgcn_perm(v[ks * 8 + 2 * m + 1], v[ks * 8 + 2 * m], 0x05040100u)); \
        o[m] = unpoison(__builtin_amdgcn_perm(v[ks * 8 + 2 * m + 1], v[ks * 8 + 2 * m], 0x07060302u)); \
      }                                                                        \
      __builtin_amdgcn_raw_buffer_store_b128(e, dr, (u32)(off) + (u32)lane * 16u + (u32)(ks * 2) * 1024u, 0, AUX_SYS); \
      __builtin_amdgcn_raw_buffer_store_b128(o, dr, (u32)(off) + (u32)lane * 16u + (u32)(ks * 2 + 1) * 1024u, 0, AUX_SYS); \
    }
#define LOAD_XH(v, rt, rb, kr_)                                                \
    _Pragma("unroll") for (int ks = 0; ks < 4; ++ks) {                         \
      const u32x4 q_ = __builtin_amdgcn_raw_buffer_load_b128(xr, x_lane + (u32)(rt) * X_RT + (u32)(rb) * 65536u + (u32)(kr_) * 256u + (u32)ks * 64u, 0, 0); \
      v[ks * 4] = q_[0]; v[ks * 4 + 1] = q_[1]; v[ks * 4 + 2] = q_[2]; v[ks * 4 + 3] = q_[3]; \
    }
#define STORE_XH(v, off)                                                       \
    _Pragma("unroll") for (int ks = 0; ks < 4; ++ks)                           \
      __builtin_amdgcn_raw_buffer_store_b128(u32x4{unpoison(v[ks * 4]), unpoison(v[ks * 4 + 1]), unpoison(v[ks * 4 + 2]), unpoison(v[ks * 4 + 3])}, dr, (u32)(off) + (u32)lane * 16u + (u32)ks * 1024u, 0, AUX_SYS);
#define PUSH_SETUP()                                                           \
    __builtin_amdgcn_sched_barrier(0);                                         \
    GET_CALL();                                                                \
    STAMP(10);                                                                 \
    const __amdgpu_buffer_rsrc_t dr = rsrc(dws + (L == 0 ? OWNW_OFF : parb), BUF_BYTES);
#define WAITCNT_(n) asm volatile("s_waitcnt vmcnt(" #n ")" ::: "memory")
#define WAITCNT(n) do { if ((n) == 0) WAITCNT_(0); else if ((n) == 4) WAITCNT_(4); else if ((n) == 8) WAITCNT_(8); else WAITCNT_(12); } while (0)
    if (L == 0) {
      u32 v[8][16];
#pragma unroll
      for (int t = 0; t < 4; ++t) { LOAD_WH(v[2 * t], t, sub, kr, 0); LOAD_WH(v[2 * t + 1], t, sub, kr, 1); }
      PUSH_SETUP();
#pragma unroll
      for (int t = 0; t < 4; ++t) {
        STORE_WH(v[2 * t], (u32)t * W_SLOT + (u32)(kr * 2 + sub) * 8192u);
        STORE_WH(v[2 * t + 1], (u32)t * W_SLOT + (u32)(kr * 2 + sub) * 8192u + 4096u);
      }
      drain();
      if (lane < 8) st_sys(dflag + (WF_OFF >> 2) + (lane >> 1) * 256 + (kr * 2 + sub) * 2 + (lane & 1), call);
      STAMP(1);
      POISON();
    } else {
      int t0 = 0, t1 = 1;
      if (L != 7) {
        while (par(fmask(t0) & L)) ++t0;
        t1 = t0 + 1; while (par(fmask(t1) & L)) ++t1;
      }
      const int s0 = t0 * 4 + ker_index(t0, L), s1 = t1 * 4 + ker_index(t1, L);
      const int xe_rt = 5 + pair_of(L);
      const u32x4 xs = {(u32)(size_t)p.x, (u32)((size_t)p.x >> 32) & 0xffffu, SHARD_BYTES, 0x00020000u};
      const u32x4 wsr = {(u32)(size_t)p.w, (u32)((size_t)p.w >> 32) & 0xffffu, SHARD_BYTES, 0x00020000u};
      u32x4 pa[4], pb[4];
      u32 off_a, fl_a, off_b, fl_b; bool isw_a, isw_b;
#define ALD1(dst, vo, imm) asm volatile("buffer_load_dword %0, %1, %2, 0 offen offset:" #imm : "=v"(dst) : "v"(vo), "s"(wsr) : "memory")
#define ALD4(dst, vo, imm) asm volatile("buffer_load_dwordx4 %0, %1, %2, 0 offen offset:" #imm : "=v"(dst) : "v"(vo), "s"(xs) : "memory")
#define ALD_W8(P, q, vo)                                                       \
          ALD1(P[q][0], vo, 0); ALD1(P[q][1], vo, 512); ALD1(P[q][2], vo, 1024); ALD1(P[q][3], vo, 1536); \
          ALD1(P[q + 1][0], vo, 2048); ALD1(P[q + 1][1], vo, 2560); ALD1(P[q + 1][2], vo, 3072); ALD1(P[q + 1][3], vo, 3584);
#define ALD_X(P, vo) ALD4(P[0], vo, 0); ALD4(P[1], vo, 64); ALD4(P[2], vo, 128); ALD4(P[3], vo, 192);
#define LOAD_PIECE(P, h_, off_, fl_, isw_)                                     \
      {                                                                        \
        const int hk = (h_) / 10, hj = (h_) - 10 * hk;                         \
        if (L == 7 || hj >= 8) {                                               \
          isw_ = false;                                                        \
          const int rt_ = L == 7 ? (hj >> 1) : xe_rt;                          \
          const u32 vo = x_lane + (u32)rt_ * X_RT + (u32)(hj & 1) * 65536u + (u32)hk * 256u; \
          ALD_X(P, vo);                                                        \
          off_ = (L == 7 ? XP_OFF + (u32)rt_ * X_RT : XE_OFF + (u32)(L - 1) * X_RT) + (u32)hk * 8192u + (u32)(hj & 1) * 4096u; \
          fl_ = (L == 7 ? (XPF_OFF >> 2) + (u32)rt_ * 32u : (XEF_OFF >> 2) + (u32)(L - 1) * 32u) + (u32)hk * 2u + (u32)(hj & 1); \
        } else {                                                               \
          const int tt = hj < 4 ? t0 : t1, sf = hj < 4 ? s0 : s1, cb = (hj >> 1) & 1; \
          isw_ = true;                                                         \
          const u32 vo = w_lane + (u32)tt * 128u + (u32)cb * 64u + (u32)(128 * hk + 64 * (hj & 1)) * 512u; \
          const u32 vo2 = vo + 16384u;                                         \
          ALD_W8(P, 0, vo); ALD_W8(P, 2, vo2);                                 \
          off_ = (u32)(sf - tt - 1) * W_SLOT + (u32)(hk * 2 + cb) * 8192u + (u32)(hj & 1) * 4096u; \
          fl_ = (WF_OFF >> 2) + (u32)sf * 64u + (u32)(hk * 2 + cb) * 2u + (u32)(hj & 1); \
        }                                                                      \
      }
      LOAD_PIECE(pa, idx, off_a, fl_a, isw_a);
      LOAD_PIECE(pb, 32 + idx, off_b, fl_b, isw_b);
      PUSH_SETUP();
      u32* lctr = (u32*)(myws + FLAG_OFF + LNK_OFF) + ((call % 3u) * 8u + (u32)L) * 64u;
      bool have_a = true, have_b = true;
      u32 grab = 0;
#define GRAB() if (lane == 0) asm volatile("global_atomic_add %0, %1, %2, off sc0" : "=v"(grab) : "v"(lctr), "v"(1u) : "memory")
      GRAB();
      asm volatile("s_waitcnt vmcnt(1)" ::: "memory");
#define PUSH_STEP(P, off_p, fl_p, isw_p, have_p)                               \
      {                                                                        \
        asm volatile("" : "+v"(P[0]), "+v"(P[1]), "+v"(P[2]), "+v"(P[3]));     \
        if (isw_p) {                                                           \
          _Pragma("unroll") for (int ks = 0; ks < 2; ++ks) {                   \
            u32x4 e, o;                                                        \
            _Pragma("unroll") for (int m = 0; m < 4; ++m) {                    \
              const u32 lo = P[ks * 2 + (m >> 1)][(2 * m) & 3], hi = P[ks * 2 + (m >> 1)][(2 * m + 1) & 3]; \
              e[m] = unpoison(__builtin_amdgcn_perm(hi, lo, 0x05040100u));     \
              o[m] = unpoison(__builtin_amdgcn_perm(hi, lo, 0x07060302u));     \
            }                                                                  \
            __builtin_amdgcn_raw_buffer_store_b128(e, dr, off_p + (u32)lane * 16u + (u32)(ks * 2) * 1024u, 0, AUX_SYS); \
            __builtin_amdgcn_raw_buffer_store_b128(o, dr, off_p + (u32)lane * 16u + (u32)(ks * 2 + 1) * 1024u, 0, AUX_SYS); \
          }                                                                    \
        } else {                                                               \
          _Pragma("unroll") for (int ks = 0; ks < 4; ++ks)                     \
            __builtin_amdgcn_raw_buffer_store_b128(u32x4{unpoison(P[ks][0]), unpoison(P[ks][1]), unpoison(P[ks][2]), unpoison(P[ks][3])}, \
                                                   dr, off_p + (u32)lane * 16u + (u32)ks * 1024u, 0, AUX_SYS); \
        }                                                                      \
        asm volatile("s_waitcnt vmcnt(4)" : "+v"(grab) :: "memory");           \
        const int h = 64 + (int)__builtin_amdgcn_readfirstlane(grab);          \
        have_p = more && h < 160;                                              \
        more = have_p;                                                         \
        if (have_p) {                                                          \
          GRAB();                                                              \
          LOAD_PIECE(P, h, off_p, fl_p, isw_p);                                \
          if (isw_p) asm volatile("s_waitcnt vmcnt(17)" ::: "memory");         \
          else asm volatile("s_waitcnt vmcnt(5)" ::: "memory");                \
        } else drain();                                                        \
        STAMP(1 + (it < 4 ? it : 4));                                          \
        ++it;                                                                  \
      }
      bool more = true;
      int it = 0;
      for (;;) {
        PUSH_STEP(pa, off_a, fl_a, isw_a, have_a);
        if (!have_b) break;
        PUSH_STEP(pb, off_b, fl_b, isw_b, have_b);
        if (!have_a) break;
      }
      STAMP(6);
      POISON();
    }
    return;
  }

  GET_CALL();
  const u32* myflag = (const u32*)(myws + FLAG_OFF);
  if (c < 8 && threadIdx.x == 0)
    __hip_atomic_store((u32*)(myws + FLAG_OFF + LNK_OFF) + (((call + 1u) % 3u) * 8u + (u32)c) * 64u, 0u,
                       __ATOMIC_RELAXED, __HIP_MEMORY_SCOPE_AGENT);
  const int xcd = c & 7, li = c >> 3;
  const int ss = 2 * xcd + (li >> 4);
  const int ti = li & 15;
  const int t = ss >> 2;
  const int s = r ^ ker_elem(t, ss & 3);
  const bool w_own = (ss & 3) == 0;
  const bool x_own = ti < 8;
  int prow, rt;
  u32 x_base, x_lane, x_rb, x_ks, x_kr;
  const u32* xflag = nullptr;
  __amdgpu_buffer_rsrc_t xr;
  if (x_own) {
    prow = r; rt = ti;
    xr = rsrc(p.x, SHARD_BYTES);
    x_base = (u32)rt * X_RT; x_lane = (u32)l16 * 4096u + (u32)g * 16u; x_rb = 65536u; x_ks = 64u; x_kr = 256u;
  } else {
    xr = rsrc(myws + parb, BUF_BYTES);
    x_lane = (u32)lane * 16u; x_rb = 4096u; x_ks = 1024u; x_kr = 8192u;
    if (ti < 13) {
      prow = r ^ 7; rt = ti - 8;
      x_base = XP_OFF + (u32)rt * X_RT;
      xflag = myflag + (XPF_OFF >> 2) + rt * 32;
    } else {
      int e = 0, n = ti - 13;
      for (int q = 1; q <= 6; ++q) if (par(fmask(t) & q)) { if (n == 0) { e = q; } --n; }
      prow = r ^ e; rt = 5 + pair_of(e);
      x_base = XE_OFF + (u32)(e - 1) * X_RT;
      xflag = myflag + (XEF_OFF >> 2) + (e - 1) * 32;
    }
  }
  const u32 w_off = w_own ? OWNW_OFF + (u32)t * W_SLOT : parb + (u32)(ss - t - 1) * W_SLOT;
  const __amdgpu_buffer_rsrc_t wr = rsrc(myws + w_off, W_SLOT);
  const u32* wflag = myflag + (WF_OFF >> 2) + ss * 64;

  const u32* pollp = nullptr;
  bool poll_flag = false;
  {
    const int pst = lane >> 3, pc = lane & 7, pkr = 4 * pst + wv;
    if (lane < 32 && pc < 2) {
      if (!x_own) {
        pollp = (const u32*)(myws + parb + x_base + (u32)pkr * 8192u + (u32)pc * 4096u + 4092u);
      }
    } else if (lane < 32 && pc < 6) {
      poll_flag = w_own;
      pollp = poll_flag ? wflag + pkr * 4 + (pc - 2)
                        : (const u32*)(myws + w_off + (u32)pkr * 16384u + (u32)(pc - 2) * 4096u + 4092u);
    }
  }
  const bool use_probe = true;

  Waiter wt; wt.tmo = tmo; wt.call = call; wt.t0 = now(); wt.spins = 0; wt.dead = false;
  u32 done = 0;

  f32x4 acc[2][4];
#pragma unroll
  for (int a = 0; a < 2; ++a)
#pragma unroll
    for (int b = 0; b < 4; ++b) acc[a][b] = f32x4{0.f, 0.f, 0.f, 0.f};
  u32x4 xb[2][8], wb[2][16];

#define LOADS(buf, kr_, AUX)                                                  \
  {                                                                           \
    if (x_own) {                                                              \
      _Pragma("unroll") for (int i = 0; i < 8; ++i)                           \
        xb[buf][i] = __builtin_amdgcn_raw_buffer_load_b128(                   \
            xr, x_base + x_lane + (kr_) * x_kr + (u32)(i >> 2) * x_rb + (u32)(i & 3) * x_ks, 0, 0); \
    } else {                                                                  \
      _Pragma("unroll") for (int i = 0; i < 8; ++i)                           \
        xb[buf][i] = __builtin_amdgcn_raw_buffer_load_b128(                   \
            xr, x_base + x_lane + (kr_) * x_kr + (u32)(i >> 2) * x_rb + (u32)(i & 3) * x_ks, 0, AUX); \
    }                                                                         \
    _Pragma("unroll") for (int i = 0; i < 16; ++i)                            \
      wb[buf][i] = __builtin_amdgcn_raw_buffer_load_b128(                     \
          wr, (u32)lane * 16u + (kr_) * 16384u + (u32)i * 1024u, 0, AUX);     \
  }
#define DBG_POLL(first)
#define POLL() { const int lo_ = __builtin_ctz(~done); \
    pv = (lane >> 3) >= lo_ + POLL_WIN ? (poll_flag ? call - 1u : 0u) : pollp ? ld_sys(pollp) : (poll_flag ? call : 1u); }
#define ISSUE(buf, first)                                                     \
  {                                                                           \
    u32 avail;                                                                \
    for (;;) {                                                                \
      if (first) POLL();                                                      \
      const bool ok_ = poll_flag ? ((int)(pv - call) >= 0) : (pv != 0u); \
      const u32 m_ = (u32)__builtin_amdgcn_ballot_w64(ok_);                   \
      DBG_POLL(first);                                                        \
      avail = 0;                                                              \
      _Pragma("unroll") for (int q = 0; q < 4; ++q)                           \
        if (((m_ >> (8 * q)) & 0x3fu) == 0x3fu) avail |= 1u << q;             \
      avail &= ~done;                                                         \
      if (avail) break;                                                       \
      if (wt.expired()) { avail = ~done & 0xfu; break; }                      \
      __builtin_amdgcn_s_sleep(POLL_SLEEP);                                   \
      if (!(first)) POLL();                                                   \
    }                                                                         \
    const u32 st_ = (u32)__builtin_ctz(avail);                                \
    done |= 1u << st_;                                                        \
    krb[buf] = 4u * st_ + (u32)wv;                                            \
    LOADS(buf, krb[buf], BULK_AUX);                                           \
    if (done != 0xfu) POLL();                                                 \
  }
  u32 pv = 0, krb[2];
#define VERIFY(buf)                                                           \
  if (use_probe) {                                                            \
    for (;;) {                                                                \
      u32 mx_ = ~0u, mw_ = ~0u;                                               \
      _Pragma("unroll") for (int i = 0; i < 8; ++i)                           \
      _Pragma("unroll") for (int q = 0; q < 4; ++q) mx_ = pk_min(mx_, xb[buf][i][q]); \
      _Pragma("unroll") for (int i = 0; i < 16; ++i)                          \
      _Pragma("unroll") for (int q = 0; q < 4; ++q) mw_ = pk_min(mw_, wb[buf][i][q]); \
      if (x_own) mx_ = ~0u;                                                   \
      const u32 m_ = pk_min(mx_, mw_);                                        \
      const bool bad_ = (m_ & 0xffffu) == 0u || (m_ >> 16) == 0u;             \
      if (__builtin_amdgcn_ballot_w64(bad_) == 0) break;                      \
      if (wt.expired()) break;                                                \
      __builtin_amdgcn_s_sleep(POLL_SLEEP);                                   \
      LOADS(buf, krb[buf], AUX_SYS);                                          \
    }                                                                         \
  }
#define COMPUTE(buf)                                                          \
  {                                                                           \
    _Pragma("unroll") for (int ks = 0; ks < 4; ++ks)                          \
    _Pragma("unroll") for (int rb = 0; rb < 2; ++rb)                          \
    _Pragma("unroll") for (int cf = 0; cf < 4; ++cf)                          \
      acc[rb][cf] = __builtin_amdgcn_mfma_f32_16x16x32_bf16(                  \
          __builtin_bit_cast(bf16x8, wb[buf][(cf >> 1) * 8 + ks * 2 + (cf & 1)]), \
          __builtin_bit_cast(bf16x8, xb[buf][rb * 4 + ks]), acc[rb][cf], 0, 0, 0); \
  }

  ISSUE(0, true);
  STAMP(1);
  VERIFY(0);
  ISSUE(1, false);
  COMPUTE(0);
  STAMP(2);
  VERIFY(1);
  ISSUE(0, false);
  COMPUTE(1);
  STAMP(3);
  VERIFY(0);
  ISSUE(1, false);
  COMPUTE(0);
  STAMP(4);
  VERIFY(1);
  COMPUTE(1);
  STAMP(5);

  {
    float* my = red + wv * 32 * LDS_ROW;
#pragma unroll
    for (int rb = 0; rb < 2; ++rb)
#pragma unroll
      for (int cb = 0; cb < 2; ++cb) {
        float* q = my + (16 * rb + l16) * LDS_ROW + 32 * cb + 8 * g;
        const f32x4 e = acc[rb][cb * 2], o = acc[rb][cb * 2 + 1];
        *(f32x4*)q = f32x4{e[0], o[0], e[1], o[1]};
        *(f32x4*)(q + 4) = f32x4{e[2], o[2], e[3], o[3]};
      }
  }
  if (lane == 0) deadf[wv] = wt.dead ? 1u : 0u;
  __syncthreads();
  STAMP(6);
  if (threadIdx.x == 0) __hip_atomic_store(ctr, call, __ATOMIC_RELAXED, __HIP_MEMORY_SCOPE_AGENT);
  const u32 tag = (call & 1u) ? 0x80008000u : 0u;
  const __amdgpu_buffer_rsrc_t myout = rsrc(p.myout, SHARD_BYTES);
  const int row = 8 * wv + (lane >> 3), col = (lane & 7) * 8;
  {
    const float* q = red + row * LDS_ROW + col;
    f32x4 a = *(const f32x4*)q, b = *(const f32x4*)(q + 4);
#pragma unroll
    for (int w2 = 1; w2 < 4; ++w2) {
      a += *(const f32x4*)(q + w2 * 32 * LDS_ROW);
      b += *(const f32x4*)(q + w2 * 32 * LDS_ROW + 4);
    }
    float y[8] = {a[0], a[1], a[2], a[3], b[0], b[1], b[2], b[3]};
    u32 h[8];
#pragma unroll
    for (int i = 0; i < 8; ++i) {
      const float v = y[i] < 0.f ? 0.f : y[i];
      const u32 u = __builtin_bit_cast(u32, v);
      h[i] = (v != v) ? 0x7fc0u : (((u + 0x7fffu + ((u >> 16) & 1u)) >> 16) & 0x7fffu);
    }
    const u32 anydead = deadf[0] | deadf[1] | deadf[2] | deadf[3];
    u32x4 o;
    if (anydead) o = u32x4{0x7fc07fc0u, 0x7fc07fc0u, 0x7fc07fc0u, 0x7fc07fc0u};
    else o = u32x4{h[0] | (h[1] << 16), h[2] | (h[3] << 16), h[4] | (h[5] << 16), h[6] | (h[7] << 16)};
    if (s == r) {
      __builtin_amdgcn_raw_buffer_store_b128(o, myout, (u32)(256 * prow + 32 * rt + row) * 512u + (u32)(64 * t + col) * 2u, 0, 0);
    } else {
      const __amdgpu_buffer_rsrc_t orr = rsrc(p.ws[s] + OUTS_OFF, 1u << 20);
      __builtin_amdgcn_raw_buffer_store_b128(o | tag, orr, (u32)((8 * prow + rt) * 4 + t) * 4096u + (u32)threadIdx.x * 16u, 0, AUX_SYS);
    }
  }
  STAMP(7);
  {
    const int ft = c & 3, frt = (c >> 2) & 7, fp = c >> 5;
    const int fd = fp ^ r;
    const bool local = par(fmask(ft) & fd) == 0 ? (fd == 0)
                       : frt < 5 ? (fd == 7) : (fd != 7 && pair_of(fd) == frt - 5);
    if (!local) {
      const __amdgpu_buffer_rsrc_t ir = rsrc(myws + OUTS_OFF, 1u << 20);
      u32x4 v;
      for (;;) {
        v = __builtin_amdgcn_raw_buffer_load_b128(ir, (u32)c * 4096u + (u32)threadIdx.x * 16u, 0, AUX_SYS);
        const u32 bad = ((v[0] ^ tag) | (v[1] ^ tag) | (v[2] ^ tag) | (v[3] ^ tag)) & 0x80008000u;
        if (__builtin_amdgcn_ballot_w64(bad != 0u) == 0) { v &= 0x7fff7fffu; break; }
        if (wt.expired()) { v = u32x4{0x7fc07fc0u, 0x7fc07fc0u, 0x7fc07fc0u, 0x7fc07fc0u}; break; }
        __builtin_amdgcn_s_sleep(POLL_SLEEP);
      }
      __builtin_amdgcn_raw_buffer_store_b128(v, myout, (u32)(256 * fp + 32 * frt + row) * 512u + (u32)(64 * ft + col) * 2u, 0, 0);
    }
  }
  STAMP(8);
}

}

extern "C" void kernel_launch(int rank, int nranks, ncclComm_t comm,
                              hipStream_t stream,
                              void* const* d_in, const int* in_sizes, int n_in,
                              void* d_out, int out_size,
                              const int* in_dtypes, int out_dtype,
                              void* const* d_out_all,
                              void* const* d_ws_all, size_t ws_bytes) {
  (void)comm; (void)in_dtypes; (void)out_dtype; (void)d_out;
  if (nranks != NR || n_in != 2 || in_sizes[0] != 256 * 2048 || in_sizes[1] != 2048 * 256 ||
      out_size != 2048 * 256 || ws_bytes < (size_t)FLAG_OFF + FLAG_BYTES) {
    fprintf(stderr, "kernel_launch: unexpected problem shape\n");
    return;
  }
  Params p;
  for (int i = 0; i < NR; ++i) { p.ws[i] = (char*)d_ws_all[i]; p.out[i] = (char*)d_out_all[i]; }
  p.x = (const char*)d_in[0];
  p.w = (const char*)d_in[1];
  p.rank = rank;
  p.myws = p.ws[rank];
  p.myout = p.out[rank];
  ag_gemm_kernel<<<256, 320, 0, stream>>>(p);
  HIP_CHECK(hipGetLastError());
}
```
